```python
import numpy as np

kernel_name = "gpu_gemm_ar_m4096_k4096_n4096_f32_gelu"


ABSMAX_RTOL = 0.001

COMPUTE_DTYPE = "float32"
EPILOGUE = "gelu"
IN_QUANT = None
OUT_QUANT = None
IN_DTYPES = {"x": "float32", "w_mat": "float32"}
WORLD_SIZE = 8

def setup_inputs():

    x = np.random.randn(4096, 4096).astype(np.float32)
    w_mat = (np.random.randn(4096, 4096) / np.sqrt(4096)).astype(np.float32)
    return {"x": x, "w_mat": w_mat}

def _epilogue(y):
    c = 0.7978845608028654
    return (0.5 * y * (1.0 + np.tanh(c * (y + 0.044715 * y ** 3)))).astype(np.float32)

def reference(x, w_mat):
    y = (x @ w_mat).astype(np.float32)
    return _epilogue(y)
```

```cpp
#include <hip/hip_runtime.h>
#include <rccl/rccl.h>
#include <cstdio>
#include <cstdint>

#define HIP_CHECK(expr) do { hipError_t _e = (expr); if (_e != hipSuccess) { \
    fprintf(stderr, "HIP error %s at %s:%d\n", hipGetErrorString(_e), __FILE__, __LINE__); \
    return; } } while (0)

namespace {

typedef unsigned u32x4 __attribute__((ext_vector_type(4)));
typedef float f32x4 __attribute__((ext_vector_type(4)));
typedef float f32x2 __attribute__((ext_vector_type(2)));
typedef _Float16 f16x8 __attribute__((ext_vector_type(8)));
typedef _Float16 f16x2 __attribute__((ext_vector_type(2)));
#define LAS __attribute__((address_space(3)))
#define GAS __attribute__((address_space(1)))

constexpr int NR = 8;
constexpr int MM = 4096, NN = 4096, KS = 512;
constexpr int NWG = 256, NTHREADS = 512;
constexpr int GEMM_PER_XCD = 26;
constexpr int NGEMM = 8 * GEMM_PER_XCD, NCOMM = NWG - NGEMM;
constexpr int NGROUPS = 1024;
constexpr int GPW = 5;
#define PULL_NW 10
constexpr unsigned long long DEADLINE = 30000000ull;

constexpr size_t WS_CALL = 0;
constexpr size_t WS_TMO = 4096;
constexpr size_t WS_SRCFLAG = 8192;
constexpr size_t WS_CNT = 16384;
constexpr size_t WS_STAMP = 65536;
constexpr size_t WS_EXPORT = 1u << 20;
constexpr size_t EXPORT_BYTES = 8u << 20;
constexpr size_t WS_RAW = WS_EXPORT + 2 * EXPORT_BYTES;
constexpr size_t RAW_BYTES = 64u << 20;
constexpr size_t WS_END = WS_RAW + RAW_BYTES;
constexpr unsigned TILE_BYTES = 1u << 20, W_OFF = 512u << 10, GROUP_BYTES = 8192;

constexpr int SLOT_BYTES = 36 * 1024;
constexpr int LDS_CTRL = 4 * SLOT_BYTES;
constexpr int LDS_BYTES = LDS_CTRL + 64;

struct Params {
    const float* x; const float* w; float* out;
    char* ws[NR];
    int rank; int pad;
};

__device__ __forceinline__ unsigned rfl(unsigned v) { return __builtin_amdgcn_readfirstlane(v); }
__device__ __forceinline__ __amdgpu_buffer_rsrc_t mkrsrc(const void* p, unsigned bytes) {
    unsigned long long a = (unsigned long long)p;
    a = ((unsigned long long)rfl((unsigned)(a >> 32)) << 32) | rfl((unsigned)a);
    return __builtin_amdgcn_make_buffer_rsrc((void*)a, 0, bytes, 0x00020000);
}
__device__ __forceinline__ unsigned ld_sys(const void* p) { return __hip_atomic_load((const GAS unsigned*)p, __ATOMIC_RELAXED, __HIP_MEMORY_SCOPE_SYSTEM); }
__device__ __forceinline__ unsigned ld_agent(const void* p) { return __hip_atomic_load((const GAS unsigned*)p, __ATOMIC_RELAXED, __HIP_MEMORY_SCOPE_AGENT); }
__device__ __forceinline__ void st_sys(void* p, unsigned v) { __hip_atomic_store((GAS unsigned*)p, v, __ATOMIC_RELAXED, __HIP_MEMORY_SCOPE_SYSTEM); }
__device__ __forceinline__ void st_agent(void* p, unsigned v) { __hip_atomic_store((GAS unsigned*)p, v, __ATOMIC_RELAXED, __HIP_MEMORY_SCOPE_AGENT); }
__device__ __forceinline__ unsigned long long now() { return __builtin_amdgcn_s_memrealtime(); }
#define WAIT_V(n) asm volatile("s_waitcnt vmcnt(" #n ")" ::: "memory")
#define WAIT_V0() do { __builtin_amdgcn_s_waitcnt(0x0F70); asm volatile("s_waitcnt vmcnt(0)" ::: "memory"); } while (0)
#define WAIT_L(n) asm volatile("s_waitcnt lgkmcnt(" #n ")" ::: "memory")
#define BAR() do { asm volatile("" ::: "memory"); __builtin_amdgcn_s_barrier(); asm volatile("" ::: "memory"); } while (0)
#define SCHED() __builtin_amdgcn_sched_barrier(0)

#define STAMP_AT(slot) do {} while (0)

__device__ __forceinline__ void order(unsigned q, unsigned& c, unsigned& isw, unsigned& g) { c = q >> 7; isw = q & 1; g = (q & 127) >> 1; }
__device__ __forceinline__ unsigned group_off(unsigned c, unsigned isw, unsigned g) { return c * TILE_BYTES + isw * W_OFF + g * GROUP_BYTES; }
__device__ __forceinline__ unsigned cnt_idx(unsigned c, unsigned isw, unsigned g, unsigned sender) {
    const unsigned panel = isw ? 13u + (g >> 2) : g / 5u;
    return panel * 64u + c * 8u + sender;
}
__device__ __forceinline__ unsigned pk(float a, float b) { f32x2 v = {a, b}; return __builtin_bit_cast(unsigned, __builtin_convertvector(v, f16x2)); }

__device__ __forceinline__ float gelu(float y) {
    const float k1 = -2.0f * 0.7978845608028654f * 1.4426950408889634f, k2 = k1 * 0.044715f;
    const float e = __builtin_amdgcn_exp2f(y * __builtin_fmaf(y * y, k2, k1));
    return y * __builtin_amdgcn_rcpf(1.0f + e);
}

__global__ void __launch_bounds__(NTHREADS, 2) fused(Params P) {
    extern __shared__ __attribute__((aligned(16))) char smem[];
    LAS char* lds = (LAS char*)smem;
    const unsigned tid = threadIdx.x, lane = tid & 63, wid = rfl(tid >> 6);
    const unsigned bid = blockIdx.x, xcd = bid & 7, loc = bid >> 3;
    const unsigned rank = (unsigned)P.rank;
    char* const ws = P.ws[rank];
    STAMP_AT(0);
    const unsigned callv = ld_agent(ws + WS_CALL + bid * 4);

    if (loc >= GEMM_PER_XCD) {
        const unsigned call = rfl(callv) + 1u, parity = call & 1u;
        const unsigned cw = (loc - GEMM_PER_XCD) * 8 + xcd;
        const unsigned gw = wid * NCOMM + cw;
        BAR();
        if (tid == 0) st_agent(ws + WS_CALL + bid * 4, call);
        if (gw >= 7u * PULL_NW) return;
        const unsigned peer = (rank + 1u + gw % 7u) & 7u, idx = gw / 7u;
        char* pws = P.ws[0];
#pragma unroll
        for (int i = 1; i < NR; ++i) pws = (peer == (unsigned)i) ? P.ws[i] : pws;
        const __amdgpu_buffer_rsrc_t rsrc = mkrsrc(pws + WS_EXPORT + parity * EXPORT_BYTES, (unsigned)EXPORT_BYTES);
        const __amdgpu_buffer_rsrc_t rdst = mkrsrc(ws + WS_RAW, (unsigned)RAW_BYTES);
        const char* sflag = ws + WS_SRCFLAG + peer * 1024;
        const unsigned nmine = (NGROUPS - idx + PULL_NW - 1) / PULL_NW;
        unsigned long long rdy0 = 0, rdy1 = 0;
        bool dead = false;
        auto wait_src = [&](unsigned j) {
            if (((j < 64 ? rdy0 : rdy1) >> (j & 63)) & 1ull) return;
            const unsigned long long t0 = now();
            for (unsigned it = 1;; ++it) {
                unsigned qa = idx + lane * PULL_NW, qb = idx + (lane + 64) * PULL_NW;
                qa = qa < NGROUPS ? qa : 0; qb = qb < NGROUPS ? qb : 0;
                const unsigned fa = ld_sys(sflag + (qa / GPW) * 4), fb = ld_sys(sflag + (qb / GPW) * 4);
                rdy0 = __ballot(fa >= call); rdy1 = __ballot(fb >= call);
                if (((j < 64 ? rdy0 : rdy1) >> (j & 63)) & 1ull) break;
                if (!(it & 15) && (rfl(ld_agent(ws + WS_TMO)) == call || now() - t0 > DEADLINE)) { if (lane == 0) st_agent(ws + WS_TMO, call); dead = true; break; }
                __builtin_amdgcn_s_sleep(1);
            }
        };
        auto load = [&](u32x4 (&R)[8], unsigned j) {
            unsigned c, isw, g; order(idx + j * PULL_NW, c, isw, g);
            const unsigned so = group_off(c, isw, g);
#pragma unroll
            for (int v = 0; v < 8; ++v) R[v] = __builtin_amdgcn_raw_buffer_load_b128(rsrc, lane * 16 + v * 1024, so, 17);
        };
        auto store = [&](u32x4 (&R)[8], unsigned j) {
            unsigned c, isw, g; order(idx + j * PULL_NW, c, isw, g);
            const unsigned so = (c * 8 + peer) * TILE_BYTES + isw * W_OFF + g * GROUP_BYTES;
#pragma unroll
            for (int v = 0; v < 8; ++v) __builtin_amdgcn_raw_buffer_store_b128(R[v], rdst, lane * 16 + v * 1024, so, 17);
        };
        auto signal = [&](unsigned j) {
            unsigned c, isw, g; order(idx + j * PULL_NW, c, isw, g);
            if (lane == 0) __hip_atomic_fetch_add((GAS unsigned*)(ws + WS_CNT) + cnt_idx(c, isw, g, peer), 1u, __ATOMIC_RELAXED, __HIP_MEMORY_SCOPE_AGENT);
        };
        u32x4 A[8], B[8];
        wait_src(0);
        STAMP_AT(1);
        if (dead) return;
        load(A, 0);
        for (unsigned j = 0;; j += 2) {
            WAIT_V0();
            if (j + 1 < nmine) { wait_src(j + 1); if (dead) return; load(B, j + 1); }
            if (j > 0) signal(j - 1);
            store(A, j);
            if (j + 1 >= nmine) { WAIT_V0(); signal(j); break; }
            WAIT_V0();
            if (j + 2 < nmine) { wait_src(j + 2); if (dead) return; load(A, j + 2); }
            signal(j);
            store(B, j + 1);
            if (j + 2 >= nmine) { WAIT_V0(); signal(j + 1); break; }
        }
        STAMP_AT(2);
        return;
    }

    const unsigned fr = lane & 15, fq = lane >> 4;
    unsigned call;
    char* exp_base;
    {
        const unsigned gi = loc * 8 + xcd;
        const unsigned q = gi * GPW + wid;
        unsigned c = 0, isw = 0, g = 0;
        const bool have = wid < GPW && q < NGROUPS;
        u32x4 V[8];
        if (have) {
            order(q, c, isw, g);
            f32x4 L[16];
            if (!isw) {
                const __amdgpu_buffer_rsrc_t rx = mkrsrc(P.x, MM * KS * 4);
                const unsigned vo = (fr * KS + fq * 4) * 4, so = (g * 64 * KS + c * 64) * 4;
#pragma unroll
                for (int b = 0; b < 4; ++b)
#pragma unroll
                    for (int u = 0; u < 4; ++u) L[b * 4 + u] = __builtin_bit_cast(f32x4, __builtin_amdgcn_raw_buffer_load_b128(rx, vo + u * 64, so + b * 16 * KS * 4, 2));
                SCHED();
#pragma unroll
                for (int h = 0; h < 2; ++h)
#pragma unroll
                    for (int b = 0; b < 4; ++b) {
                        const f32x4 lo = L[b * 4 + 2 * h], hi = L[b * 4 + 2 * h + 1];
                        V[h * 4 + b] = (u32x4){pk(lo[0], lo[1]), pk(lo[2], lo[3]), pk(hi[0], hi[1]), pk(hi[2], hi[3])};
                    }
            } else {
                const __amdgpu_buffer_rsrc_t rw = mkrsrc(P.w, KS * NN * 4);
                const unsigned vo = (fq * 4 * NN + fr * 4) * 4, so = (c * 64 * NN + g * 64) * 4;
#pragma unroll
                for (int u = 0; u < 4; ++u)
#pragma unroll
                    for (int j = 0; j < 4; ++j) L[u * 4 + j] = __builtin_bit_cast(f32x4, __builtin_amdgcn_raw_buffer_load_b128(rw, vo, so + (u * 16 + j) * NN * 4, 2));
                SCHED();
#pragma unroll
                for (int h = 0; h < 2; ++h)
#pragma unroll
                    for (int cc = 0; cc < 4; ++cc) {
                        const f32x4* l0 = &L[(2 * h) * 4]; const f32x4* l1 = &L[(2 * h + 1) * 4];
                        V[h * 4 + cc] = (u32x4){pk(l0[0][cc], l0[1][cc]), pk(l0[2][cc], l0[3][cc]), pk(l1[0][cc], l1[1][cc]), pk(l1[2][cc], l1[3][cc])};
                    }
            }
        }
        SCHED();
        call = rfl(callv) + 1u;
        exp_base = ws + WS_EXPORT + (call & 1u) * EXPORT_BYTES;
        if (have) {
            const __amdgpu_buffer_rsrc_t rexp = mkrsrc(exp_base, (unsigned)EXPORT_BYTES);
            const unsigned so = group_off(c, isw, g);
#pragma unroll
            for (int v = 0; v < 8; ++v) __builtin_amdgcn_raw_buffer_store_b128(V[v], rexp, lane * 16 + v * 1024, so, 17);
        }
        WAIT_V(0);
        if (have && lane == 0) __hip_atomic_fetch_add((GAS unsigned*)(ws + WS_CNT) + cnt_idx(c, isw, g, rank), 1u, __ATOMIC_RELAXED, __HIP_MEMORY_SCOPE_AGENT);
        BAR();
        if (wid == 0) {
            if (lane < NR && lane != rank) {
                char* pws = P.ws[0];
#pragma unroll
                for (int i = 1; i < NR; ++i) pws = (lane == (unsigned)i) ? P.ws[i] : pws;
                st_sys(pws + WS_SRCFLAG + rank * 1024 + gi * 4, call);
            }
            if (lane == 0) st_agent(ws + WS_CALL + bid * 4, call);
        }
    }
    STAMP_AT(1);

    const unsigned L = (xcd & 1) * GEMM_PER_XCD + loc, pm = L >> 2, pn = (xcd >> 1) * 4 + (L & 3);
    const unsigned wr = wid >> 2, wc = wid & 3;
    unsigned voff0, voff1;
    {
        unsigned ga = pm * 5 + wid; ga = ga > 63 ? 63 : ga;
        voff0 = (wid < 5 ? ga * GROUP_BYTES : W_OFF + (pn * 4 + wid - 5) * GROUP_BYTES) + lane * 16;
        voff1 = W_OFF + (pn * 4 + 3) * GROUP_BYTES + wc * 1024 + lane * 16;
    }
    const unsigned lds0 = (unsigned)(uintptr_t)lds;
    const unsigned a_rd = lane * 16 + wr * 10240, b_rd = lane * 16 + 20480 + wc * 4096;
    const unsigned needx = call * (pm == 12 ? 4u : 5u), needw = call * 4u;
    const char* const cntx = ws + WS_CNT + (pm * 64 + lane) * 4;
    const char* const cntw = ws + WS_CNT + ((13 + pn) * 64 + lane) * 4;
    LAS unsigned long long* const ctrl = (LAS unsigned long long*)(lds + LDS_CTRL);

    unsigned long long avail = 0, taken = 0;
    unsigned long long tbase = 0;
    unsigned tail = 0;

    auto poll = [&]() {
        const unsigned long long t0 = now();
        unsigned long long m;
        for (unsigned it = 1;; ++it) {
            const unsigned cx = ld_agent(cntx), cwv = ld_agent(cntw);
            m = __ballot(cx >= needx && cwv >= needw) & ~taken;
            if (m) break;
            if (!(it & 15) && (rfl(ld_agent(ws + WS_TMO)) == call || now() - t0 > DEADLINE)) { if (lane == 0) st_agent(ws + WS_TMO, call); m = ~taken; break; }
            __builtin_amdgcn_s_sleep(2);
        }
        if (lane == 0) *ctrl = m;
        WAIT_L(0);
    };
    auto fetch_mask = [&]() {
        const unsigned long long m = *(volatile LAS unsigned long long*)ctrl;
        avail = ((unsigned long long)rfl((unsigned)(m >> 32)) << 32) | rfl((unsigned)m);
    };
    auto stage_upto = [&](unsigned lim) {
        while (tail < lim) {
            if (!(tail & 1)) {
                if (!avail) break;
                const unsigned t = __builtin_ctzll(avail);
                avail &= avail - 1; taken |= 1ull << t;
                tbase = (t & 7) == rank ? (unsigned long long)exp_base + (t >> 3) * TILE_BYTES : (unsigned long long)(ws + WS_RAW) + t * TILE_BYTES;
            } else tbase += 4096;
            const unsigned m0a = lds0 + (tail & 3) * SLOT_BYTES + wid * 4096;
            unsigned keep;
            asm volatile("s_mov_b32 %0, m0\n\ts_mov_b32 m0, %3\n\ts_nop 0\n\t"
                         "global_load_lds_dwordx4 %1, %2 sc0 sc1\n\t"
                         "global_load_lds_dwordx4 %1, %2 offset:1024 sc0 sc1\n\t"
                         "global_load_lds_dwordx4 %1, %2 offset:2048 sc0 sc1\n\t"
                         "global_load_lds_dwordx4 %1, %2 offset:3072 sc0 sc1\n\t"
                         "s_mov_b32 m0, %0" : "=&s"(keep) : "v"(voff0), "s"(tbase), "s"(m0a) : "memory");
            if (wr == 0) {
                const unsigned m0b = lds0 + (tail & 3) * SLOT_BYTES + 32768 + wc * 1024;
                asm volatile("s_mov_b32 %0, m0\n\ts_mov_b32 m0, %3\n\ts_nop 0\n\t"
                             "global_load_lds_dwordx4 %1, %2 sc0 sc1\n\t"
                             "s_mov_b32 m0, %0" : "=&s"(keep) : "v"(voff1), "s"(tbase), "s"(m0b) : "memory");
            }
            ++tail;
        }
    };
    auto wait_steps = [&](unsigned n) {
        if (n >= 2) { if (wr == 0) WAIT_V(10); else WAIT_V(8); }
        else if (n == 1) { if (wr == 0) WAIT_V(5); else WAIT_V(4); }
        else WAIT_V(0);
    };

    f32x4 acc[10][4];
#pragma unroll
    for (int m = 0; m < 10; ++m)
#pragma unroll
        for (int n = 0; n < 4; ++n) acc[m][n] = (f32x4){0.f, 0.f, 0.f, 0.f};

    WAIT_V(0);
    if (wid == 0) poll();
    BAR();
    fetch_mask();
    stage_upto(3);
    wait_steps(tail - 1);
    BAR();
    if (wr == 1) BAR();
    STAMP_AT(2);

    constexpr unsigned NSTEP = 128;
    for (unsigned s = 0; s < NSTEP; ++s) {
        const unsigned sb = (s & 3) * SLOT_BYTES;
        const LAS char* pa = lds + sb + a_rd; const LAS char* pb = lds + sb + b_rd;
        f16x8 At[5], Bt[4];
#pragma unroll
        for (int n = 0; n < 4; ++n) Bt[n] = *(const LAS f16x8*)(pb + n * 1024);
#pragma unroll
        for (int m = 0; m < 5; ++m) At[m] = *(const LAS f16x8*)(pa + m * 1024);
        BAR(); WAIT_L(0); SCHED();
        __builtin_amdgcn_s_setprio(1);
#pragma unroll
        for (int m = 0; m < 5; ++m)
#pragma unroll
            for (int n = 0; n < 4; ++n) acc[m][n] = __builtin_amdgcn_mfma_f32_16x16x32_f16(At[m], Bt[n], acc[m][n], 0, 0, 0);
        __builtin_amdgcn_s_setprio(0);
        SCHED(); BAR();
#pragma unroll
        for (int m = 0; m < 5; ++m) At[m] = *(const LAS f16x8*)(pa + (5 + m) * 1024);
        const unsigned lim = s + 4 < NSTEP ? s + 4 : NSTEP;
        stage_upto(lim);
        if (tail == s + 1 && s + 1 < NSTEP) {
            if (wr == 0) { if (wid == 0) poll(); BAR(); }
            fetch_mask();
            stage_upto(lim);
            wait_steps(tail - (s + 2));
            if (wr == 1) BAR();
        } else if (s + 1 < NSTEP) wait_steps(tail - (s + 2));
        BAR(); WAIT_L(0); SCHED();
        __builtin_amdgcn_s_setprio(1);
#pragma unroll
        for (int m = 0; m < 5; ++m)
#pragma unroll
            for (int n = 0; n < 4; ++n) acc[5 + m][n] = __builtin_amdgcn_mfma_f32_16x16x32_f16(At[m], Bt[n], acc[5 + m][n], 0, 0, 0);
        __builtin_amdgcn_s_setprio(0);
        SCHED(); BAR();
    }
    if (wr == 0) BAR();
    STAMP_AT(3);

    const bool bad = rfl(ld_agent(ws + WS_TMO)) == call;
    const unsigned row0 = pm * 320 + wr * 160 + fq * 4;
    float* const op = P.out + (size_t)row0 * NN + pn * 256 + wc * 64 + fr * 4;
    const float qnan = __builtin_nanf("");
#pragma unroll
    for (int m = 0; m < 10; ++m) {
        if (pm * 320 + wr * 160 + m * 16 < MM) {
#pragma unroll
            for (int j = 0; j < 4; ++j) {
                f32x4 o = {gelu(acc[m][0][j]), gelu(acc[m][1][j]), gelu(acc[m][2][j]), gelu(acc[m][3][j])};
                if (bad) o = (f32x4){qnan, qnan, qnan, qnan};
                *(f32x4*)(op + (size_t)(m * 16 + j) * NN) = o;
            }
        }
    }
}

}

extern "C" void kernel_launch(int rank, int nranks, ncclComm_t comm,
                              hipStream_t stream,
                              void* const* d_in, const int* in_sizes, int n_in,
                              void* d_out, int out_size,
                              const int* in_dtypes, int out_dtype,
                              void* const* d_out_all,
                              void* const* d_ws_all, size_t ws_bytes) {
    (void)comm; (void)in_dtypes; (void)out_dtype; (void)d_out_all;
    if (nranks != NR || n_in != 2 || (unsigned)rank >= (unsigned)NR || ws_bytes < WS_END || out_size != MM * NN ||
        in_sizes[0] != MM * KS || in_sizes[1] != KS * NN) {
        fprintf(stderr, "kernel_launch: unexpected problem shape\n");
        return;
    }
    HIP_CHECK(hipFuncSetAttribute((const void*)fused, hipFuncAttributeMaxDynamicSharedMemorySize, LDS_BYTES));
    Params P;
    P.x = (const float*)d_in[0]; P.w = (const float*)d_in[1]; P.out = (float*)d_out;
    for (int i = 0; i < NR; ++i) P.ws[i] = (char*)d_ws_all[i];
    P.rank = rank; P.pad = 0;
    hipLaunchKernelGGL(fused, dim3(NWG), dim3(NTHREADS), LDS_BYTES, stream, P);
    HIP_CHECK(hipGetLastError());
}
```

```cpp
#include <hip/hip_runtime.h>
#include <rccl/rccl.h>
#include <cstdio>
#include <cstdint>

#define HIP_CHECK(expr) do { hipError_t _e = (expr); if (_e != hipSuccess) { \
    fprintf(stderr, "HIP error %s at %s:%d\n", hipGetErrorString(_e), __FILE__, __LINE__); \
    return; } } while (0)

namespace {

typedef unsigned u32x4 __attribute__((ext_vector_type(4)));
typedef float f32x4 __attribute__((ext_vector_type(4)));
typedef float f32x2 __attribute__((ext_vector_type(2)));
typedef _Float16 f16x8 __attribute__((ext_vector_type(8)));
typedef _Float16 f16x2 __attribute__((ext_vector_type(2)));
#define LAS __attribute__((address_space(3)))
#define GAS __attribute__((address_space(1)))

constexpr int NR = 8;
constexpr int MM = 4096, NN = 4096, KS = 512;
constexpr int NWG = 256, NTHREADS = 512;
constexpr int GEMM_PER_XCD = 26;
constexpr int NGEMM = 8 * GEMM_PER_XCD, NCOMM = NWG - NGEMM;
constexpr int NGROUPS = 1024;
constexpr int GPW = 5;
#define PULL_NW 12
#define HEAD_NOW 48
#define HEAD_GAP 100
#define HEAD_SLOPE 10
#define OUT_AUX 2
#define RAW_HEAD 32
#define ROT_FROM 6
#define PULL_PIPE 1
constexpr unsigned long long DEADLINE = 30000000ull;

constexpr size_t WS_CALL = 0;
constexpr size_t WS_TMO = 4096;
constexpr size_t WS_TICKET = 5120;
constexpr size_t WS_CNT = 8192;
constexpr size_t WS_SRCFLAG = 16384;
constexpr size_t WS_STAMP = 65536;
constexpr size_t WS_TRACE = 524288;
constexpr size_t WS_RAW = 2u << 20;
constexpr size_t RAW_BYTES = 64u << 20;
constexpr unsigned PACK_STRIDE = 7424;
constexpr size_t WS_PACK = WS_RAW + 2 * RAW_BYTES;
constexpr size_t PACK_BYTES = 8u << 20;
constexpr size_t WS_END = WS_PACK + 2 * PACK_BYTES;
constexpr unsigned TILE_BYTES = 1u << 20, W_OFF = 512u << 10, GROUP_BYTES = 8192;

constexpr int SLOT_BYTES = 36 * 1024;
constexpr int LDS_CTRL = 4 * SLOT_BYTES;
constexpr int LDS_SNAP = LDS_CTRL + 64;
constexpr int LDS_BYTES = LDS_SNAP + 1536;

struct Params {
    const float* x; const float* w; float* out;
    char* self;
    char* ws[NR];
    int rank; int pad;
};

__device__ __forceinline__ unsigned rfl(unsigned v) { return __builtin_amdgcn_readfirstlane(v); }
__device__ __forceinline__ __amdgpu_buffer_rsrc_t mkrsrc(const void* p, unsigned bytes) {
    unsigned long long a = (unsigned long long)p;
    a = ((unsigned long long)rfl((unsigned)(a >> 32)) << 32) | rfl((unsigned)a);
    return __builtin_amdgcn_make_buffer_rsrc((void*)a, 0, bytes, 0x00020000);
}
__device__ __forceinline__ unsigned ld_sys(const void* p) { return __hip_atomic_load((const GAS unsigned*)p, __ATOMIC_RELAXED, __HIP_MEMORY_SCOPE_SYSTEM); }
__device__ __forceinline__ unsigned ld_agent(const void* p) { return __hip_atomic_load((const GAS unsigned*)p, __ATOMIC_RELAXED, __HIP_MEMORY_SCOPE_AGENT); }
__device__ __forceinline__ void st_sys(void* p, unsigned v) { __hip_atomic_store((GAS unsigned*)p, v, __ATOMIC_RELAXED, __HIP_MEMORY_SCOPE_SYSTEM); }
__device__ __forceinline__ void st_agent(void* p, unsigned v) { __hip_atomic_store((GAS unsigned*)p, v, __ATOMIC_RELAXED, __HIP_MEMORY_SCOPE_AGENT); }
__device__ __forceinline__ unsigned add_agent(void* p, unsigned v) {
    unsigned z = 0; asm("" : "+v"(z));
    return __hip_atomic_fetch_add((GAS unsigned*)p + z, v, __ATOMIC_RELAXED, __HIP_MEMORY_SCOPE_AGENT);
}
__device__ __forceinline__ unsigned long long now() { return __builtin_amdgcn_s_memrealtime(); }
#define WAIT_V(n) asm volatile("s_waitcnt vmcnt(" #n ")" ::: "memory")
#define WAIT_V0() do { __builtin_amdgcn_s_waitcnt(0x0F70); asm volatile("s_waitcnt vmcnt(0)" ::: "memory"); } while (0)
#define WAIT_L(n) asm volatile("s_waitcnt lgkmcnt(" #n ")" ::: "memory")
#define BAR() do { asm volatile("" ::: "memory"); __builtin_amdgcn_s_barrier(); asm volatile("" ::: "memory"); } while (0)
#define SCHED() __builtin_amdgcn_sched_barrier(0)

#define STAMP_AT(slot) do {} while (0)

__device__ __forceinline__ void order(unsigned q, unsigned& c, unsigned& isw, unsigned& g) { c = q >> 7; isw = q & 1; g = (q & 127) >> 1; }
__device__ __forceinline__ unsigned link_order(unsigned tq, unsigned link) {
    return tq < ROT_FROM * 128u ? tq : (tq & ~127u) | ((tq + link * 18u) & 127u);
}
__device__ __forceinline__ unsigned group_off(unsigned c, unsigned isw, unsigned g, unsigned sender) { return (c * 8 + sender) * TILE_BYTES + isw * W_OFF + g * GROUP_BYTES; }
__device__ __forceinline__ unsigned cnt_idx(unsigned c, unsigned isw, unsigned g, unsigned sender) {
    const unsigned panel = isw ? 13u + (g >> 2) : g / 5u;
    return panel * 64u + c * 8u + sender;
}
__device__ __forceinline__ unsigned pk(float a, float b) { f32x2 v = {a, b}; return __builtin_bit_cast(unsigned, __builtin_convertvector(v, f16x2)); }

typedef unsigned short u16x2 __attribute__((ext_vector_type(2)));
__device__ __forceinline__ unsigned bfi(unsigned m, unsigned a, unsigned b) { return (a & m) | (b & ~m); }
__device__ __forceinline__ unsigned pk_add16(unsigned a, unsigned b) { return __builtin_bit_cast(unsigned, __builtin_bit_cast(u16x2, a) + __builtin_bit_cast(u16x2, b)); }
__device__ __forceinline__ unsigned pk_sub16(unsigned a, unsigned b) { return __builtin_bit_cast(unsigned, __builtin_bit_cast(u16x2, a) - __builtin_bit_cast(u16x2, b)); }
#define LDS_FENCE() do { asm volatile("" ::: "memory"); __builtin_amdgcn_wave_barrier(); } while (0)

__global__ void __launch_bounds__(NTHREADS, 2) fused(Params P) {
    extern __shared__ __attribute__((aligned(16))) char smem[];
    LAS char* lds = (LAS char*)smem;
    const unsigned tid = threadIdx.x, lane = tid & 63, wid = rfl(tid >> 6);
    const unsigned bid = blockIdx.x, xcd = bid & 7, loc = bid >> 3;
    const unsigned rank = (unsigned)P.rank;
    char* const ws = P.self;
    STAMP_AT(0);
    const unsigned callv = ld_agent(ws + WS_CALL + bid * 4);

    if (loc >= GEMM_PER_XCD) {
        const unsigned call = rfl(callv) + 1u, parity = call & 1u;
        const unsigned cw = (loc - GEMM_PER_XCD) * 8 + xcd;
        const unsigned gw = wid * NCOMM + cw;
        WAIT_V(0); BAR();
        if (tid == 0) st_agent(ws + WS_CALL + bid * 4, call);
        if (gw >= 7u * PULL_NW) return;
        const unsigned link = gw % 7u, peer = (rank + 1u + link) & 7u, idx = gw / 7u;
        char* pws = P.ws[0];
#pragma unroll
        for (int i = 1; i < NR; ++i) pws = (peer == (unsigned)i) ? P.ws[i] : pws;
        const unsigned long long pbase = ((unsigned long long)rfl((unsigned)((unsigned long long)pws >> 32)) << 32) | rfl((unsigned)(unsigned long long)pws);
        const __amdgpu_buffer_rsrc_t rsrc = __builtin_amdgcn_make_buffer_rsrc((void*)pbase, 0, (unsigned)WS_END, 0x00020000);
        const unsigned raw_so = (unsigned)WS_RAW + parity * (unsigned)RAW_BYTES, pak_so = (unsigned)WS_PACK + parity * (unsigned)PACK_BYTES;
        LAS unsigned* const ex = (LAS unsigned*)(lds + wid * 2048);
        const __amdgpu_buffer_rsrc_t rdst = mkrsrc(ws + WS_RAW + parity * RAW_BYTES, (unsigned)RAW_BYTES);
        const char* sflag = ws + WS_SRCFLAG + peer * 2048;
        char* const ticket = ws + WS_TICKET + peer * 128;
        const unsigned tbias = PULL_NW - (call - 1u) * NGROUPS;
        unsigned known = 0;
        bool dead = false;
        auto wait_src = [&](unsigned q) {
            if (q < known) return;
            const unsigned f = q < NGEMM ? q : 256u + q % NGEMM;
            const unsigned long long t0 = now();
            for (unsigned it = 1;; ++it) {
                unsigned v[8]; bool all1 = true, mine = false;
#pragma unroll
                for (int k = 0; k < 8; ++k) v[k] = ld_sys(sflag + (k * 64 + lane) * 4);
                unsigned pre = NGEMM;
#pragma unroll
                for (int k = 7; k >= 0; --k) {
                    const unsigned i = k * 64 + lane; const bool ok = v[k] >= call || (i & 255u) >= NGEMM;
                    if (k < 4) { const unsigned long long miss = __ballot(!ok); if (miss) pre = k * 64 + __builtin_ctzll(miss); } else all1 &= ok;
                    mine |= ok && i == f;
                }
                known = pre == NGEMM && __all(all1) ? NGROUPS : pre;
                if (__any(mine)) break;
                if (!(it & 15) && (rfl(ld_agent(ws + WS_TMO)) == call || now() - t0 > DEADLINE)) { if (lane == 0) st_agent(ws + WS_TMO, call); dead = true; break; }
                __builtin_amdgcn_s_sleep(1);
            }
        };
        auto draw = [&]() { unsigned t = 0; if (lane == 0) t = add_agent(ticket, 1u); return t; };
        auto goff = [&](unsigned q) { unsigned c, isw, g; order(q, c, isw, g); return group_off(c, isw, g, peer); };
        auto load = [&](u32x4 (&R)[8], unsigned& rec, unsigned q) {
            const bool isr = q < RAW_HEAD;
            const unsigned so = isr ? raw_so + goff(q) : pak_so + q * PACK_STRIDE;
            const __amdgpu_buffer_rsrc_t r7 = __builtin_amdgcn_make_buffer_rsrc((void*)pbase, 0, isr ? (unsigned)WS_END : 0u, 0x00020000);
            const __amdgpu_buffer_rsrc_t rr = __builtin_amdgcn_make_buffer_rsrc((void*)pbase, 0, isr ? 0u : (unsigned)WS_END, 0x00020000);
            rec = __builtin_amdgcn_raw_buffer_load_b16(rr, 7168 + lane * 2, so, 17);
#pragma unroll
            for (int v = 0; v < 7; ++v) R[v] = __builtin_amdgcn_raw_buffer_load_b128(rsrc, lane * 16 + v * 1024, so, 17);
            R[7] = __builtin_amdgcn_raw_buffer_load_b128(r7, lane * 16 + 7168, so, 17);
        };
        auto store = [&](const u32x4 (&R)[8], unsigned q) {
            const unsigned so = goff(q);
#pragma unroll
            for (int v = 0; v < 8; ++v) __builtin_amdgcn_raw_buffer_store_b128(R[v], rdst, lane * 16 + v * 1024, so, 17);
        };
        auto deliver = [&](const u32x4 (&R)[8], unsigned rec, unsigned q) {
            if (q < RAW_HEAD) { store(R, q); return; }
            rec &= 0xffffu;
            const unsigned hdr = rfl(rec), mode = hdr >> 12;
            u32x4 O[8];
            if (mode > 1) {
#pragma unroll
                for (int v = 0; v < 8; ++v) O[v] = __builtin_amdgcn_raw_buffer_load_b128(rsrc, lane * 16 + v * 1024, raw_so + goff(q), 17);
                WAIT_V0();
                store(O, q);
                return;
            }
#pragma unroll
            for (int a = 0; a < 5; ++a) ex[a * 64 + lane] = 0;
            LDS_FENCE();
            rec = lane ? rec : 0u;
            __hip_atomic_fetch_or(ex + (rec >> 6), (rec & 3u) << ((rec >> 1) & 30u), __ATOMIC_RELAXED, __HIP_MEMORY_SCOPE_WORKGROUP);
            if (mode == 1) {
                rec = __builtin_amdgcn_raw_buffer_load_b16(rsrc, 7296 + lane * 2, pak_so + q * PACK_STRIDE, 17) & 0xffffu;
                __hip_atomic_fetch_or(ex + (rec >> 6), (rec & 3u) << ((rec >> 1) & 30u), __ATOMIC_RELAXED, __HIP_MEMORY_SCOPE_WORKGROUP);
            }
            LDS_FENCE();
            unsigned E[5];
#pragma unroll
            for (int a = 0; a < 5; ++a) E[a] = ex[a * 64 + lane];
            LDS_FENCE();
            const unsigned b2 = (hdr & 0x1ffu) * 0x00400040u;
#pragma unroll
            for (int k = 0; k < 4; ++k) {
                unsigned c7 = 0;
#pragma unroll
                for (int v = 0; v < 7; ++v) c7 = bfi(0x00030003u << (2 * v), R[v][k] >> (13 - 2 * v), c7);
                const unsigned t7 = bfi(0x1fff1fffu, c7, c7 << 2);
                O[7][k] = pk_add16(bfi(0x60006000u, E[4] << (13 - 2 * k), t7), b2);
            }
#pragma unroll
            for (int v = 0; v < 7; ++v)
#pragma unroll
                for (int k = 0; k < 4; ++k) O[v][k] = pk_add16(bfi(0x60006000u, E[k] << (13 - 2 * v), R[v][k]), b2);
            store(O, q);
        };
        auto signal = [&](unsigned q) {
            unsigned c, isw, g; order(q, c, isw, g);
            if (lane == 0) add_agent(ws + WS_CNT + cnt_idx(c, isw, g, peer) * 4, 1u);
        };
        u32x4 A[8], B[8];
        unsigned ra, rb = 0;
        unsigned qa = idx, qb, qprev = ~0u;
        wait_src(qa);
        STAMP_AT(1);
        if (dead) return;
        load(A, ra, qa);
        unsigned tk = draw();
        for (;;) {
            WAIT_V0();
            qb = rfl(tk) + tbias;
            if (qb < NGROUPS) { qb = link_order(qb, link); wait_src(qb); if (dead) return; load(B, rb, qb); tk = draw(); }
            if (qprev != ~0u) signal(qprev);
            deliver(A, ra, qa);
            WAIT_V0();
            if (qb >= NGROUPS) { signal(qa); break; }
            unsigned qn = rfl(tk) + tbias;
            if (qn < NGROUPS) { qn = link_order(qn, link); wait_src(qn); if (dead) return; load(A, ra, qn); tk = draw(); }
            signal(qa);
            deliver(B, rb, qb);
            if (qn >= NGROUPS) { WAIT_V0(); signal(qb); break; }
            qprev = qb; qa = qn;
        }
        STAMP_AT(2);
        return;
    }

    const unsigned fr = lane & 15, fq = lane >> 4;
    unsigned call;
    char* raw;
    {
        const unsigned gi = loc * 8 + xcd;
        const unsigned q = wid * NGEMM + gi;
        char* pws = P.ws[0];
#pragma unroll
        for (int i = 1; i < NR; ++i) pws = ((lane & 7) == (unsigned)i) ? P.ws[i] : pws;
        asm volatile("" : "+v"(pws));
        if (q >= HEAD_NOW && q < NGROUPS) {
            const unsigned long long t0 = now(), dt = HEAD_GAP + (q * HEAD_SLOPE) / 10;
            while (now() - t0 < dt) __builtin_amdgcn_s_sleep(2);
        }
        unsigned c = 0, isw = 0, g = 0;
        const bool have = wid < GPW && q < NGROUPS;
        u32x4 V[8];
        float asum = 0.f;
        if (have) {
            order(q, c, isw, g);
            f32x4 L[16];
            if (!isw) {
                const __amdgpu_buffer_rsrc_t rx = mkrsrc(P.x, MM * KS * 4);
                const unsigned vo = (fr * KS + fq * 4) * 4, so = (g * 64 * KS + c * 64) * 4;
#pragma unroll
                for (int b = 0; b < 4; ++b)
#pragma unroll
                    for (int u = 0; u < 4; ++u) L[b * 4 + u] = __builtin_bit_cast(f32x4, __builtin_amdgcn_raw_buffer_load_b128(rx, vo + u * 64, so + b * 16 * KS * 4, 2));
                SCHED();
#pragma unroll
                for (int h = 0; h < 2; ++h)
#pragma unroll
                    for (int b = 0; b < 4; ++b) {
                        const f32x4 lo = L[b * 4 + 2 * h], hi = L[b * 4 + 2 * h + 1];
                        V[h * 4 + b] = (u32x4){pk(lo[0], lo[1]), pk(lo[2], lo[3]), pk(hi[0], hi[1]), pk(hi[2], hi[3])};
                    }
            } else {
                const __amdgpu_buffer_rsrc_t rw = mkrsrc(P.w, KS * NN * 4);
                const unsigned vo = (fq * 4 * NN + fr * 4) * 4, so = (c * 64 * NN + g * 64) * 4;
#pragma unroll
                for (int u = 0; u < 4; ++u)
#pragma unroll
                    for (int j = 0; j < 4; ++j) L[u * 4 + j] = __builtin_bit_cast(f32x4, __builtin_amdgcn_raw_buffer_load_b128(rw, vo, so + (u * 16 + j) * NN * 4, 2));
                SCHED();
#pragma unroll
                for (int h = 0; h < 2; ++h)
#pragma unroll
                    for (int cc = 0; cc < 4; ++cc) {
                        const f32x4* l0 = &L[(2 * h) * 4]; const f32x4* l1 = &L[(2 * h + 1) * 4];
                        V[h * 4 + cc] = (u32x4){pk(l0[0][cc], l0[1][cc]), pk(l0[2][cc], l0[3][cc]), pk(l1[0][cc], l1[1][cc]), pk(l1[2][cc], l1[3][cc])};
                    }
            }
#pragma unroll
            for (int i = 0; i < 16; i += 5) asum += (__builtin_fabsf(L[i][0]) + __builtin_fabsf(L[i][1])) + (__builtin_fabsf(L[i][2]) + __builtin_fabsf(L[i][3]));
        }
        SCHED();
        call = rfl(callv) + 1u;
        raw = ws + WS_RAW + (call & 1u) * RAW_BYTES;
        if (have) {
            const __amdgpu_buffer_rsrc_t rexp = mkrsrc(raw, (unsigned)RAW_BYTES);
            const unsigned so = group_off(c, isw, g, rank);
#pragma unroll
            for (int v = 0; v < 8; ++v) __builtin_amdgcn_raw_buffer_store_b128(V[v], rexp, lane * 16 + v * 1024, so, 17);
        }
        if (have && q >= RAW_HEAD) {
            int si = __builtin_bit_cast(int, asum);
#define ROW_ADD(ctl) si = __builtin_bit_cast(int, __builtin_bit_cast(float, si) + __builtin_bit_cast(float, __builtin_amdgcn_update_dpp(0, si, ctl, 0xf, 0xf, true)))
            ROW_ADD(0x111); ROW_ADD(0x112); ROW_ADD(0x114); ROW_ADD(0x118);
            const float top = __builtin_bit_cast(float, __builtin_amdgcn_readlane(si, 15)) * (4.17f / 256.f);
            int bb = (int)(pk(top, top) & 0x7fffu) - 8192;
            bb = bb < 0 ? 0 : bb;
            const unsigned b6 = rfl((unsigned)bb >> 6), b2 = b6 * 0x00400040u;
            LAS unsigned short* const rec16 = (LAS unsigned short*)(lds + wid * 512);
            ((LAS unsigned*)rec16)[lane] = 0;
            LDS_FENCE();
            unsigned E[5] = {0, 0, 0, 0, 0};
#pragma unroll
            for (int k = 0; k < 4; ++k) {
                const unsigned t7 = pk_sub16(V[7][k], b2), c7 = bfi(0x1fff1fffu, t7, t7 >> 2);
                E[4] = bfi(0x00030003u << (2 * k), t7 >> (13 - 2 * k), E[4]);
#pragma unroll
                for (int v = 0; v < 7; ++v) {
                    const unsigned t = pk_sub16(V[v][k], b2);
                    E[k] = bfi(0x00030003u << (2 * v), t >> (13 - 2 * v), E[k]);
                    V[v][k] = bfi(0x60006000u, c7 << (13 - 2 * v), t);
                }
            }
            const __amdgpu_buffer_rsrc_t rpak = mkrsrc(ws + WS_PACK + (call & 1u) * PACK_BYTES, NGROUPS * PACK_STRIDE);
#pragma unroll
            for (int v = 0; v < 7; ++v) __builtin_amdgcn_raw_buffer_store_b128(V[v], rpak, lane * 16 + v * 1024, q * PACK_STRIDE, 17);
            unsigned cnt = 1;
#pragma unroll
            for (int a = 0; a < 5; ++a) {
                unsigned e = E[a];
                while (cnt <= 128) {
                    const unsigned long long m = __ballot(e != 0);
                    if (!m) break;
                    if (e != 0) {
                        const unsigned pos = __builtin_ctz(e) & 30u;
                        const unsigned slot = cnt + __builtin_amdgcn_mbcnt_hi((unsigned)(m >> 32), __builtin_amdgcn_mbcnt_lo((unsigned)m, 0));
                        if (slot < 128) rec16[slot] = (unsigned short)(((a * 64 + lane) << 6) | (pos << 1) | ((e >> pos) & 3u));
                        e &= ~(3u << pos);
                    }
                    cnt += __builtin_popcountll(m);
                }
            }
            if (lane == 0) rec16[0] = (unsigned short)(b6 | ((cnt <= 64 ? 0u : cnt <= 128 ? 1u : 2u) << 12));
            LDS_FENCE();
            __builtin_amdgcn_raw_buffer_store_b32(((LAS unsigned*)rec16)[lane], rpak, 7168 + lane * 4, q * PACK_STRIDE, 17);
        }
        WAIT_V(0);
        if (wid == 0 && lane < NR && lane != rank) st_sys(pws + WS_SRCFLAG + rank * 2048 + gi * 4, call);
        if (have && lane == 0) add_agent(ws + WS_CNT + cnt_idx(c, isw, g, rank) * 4, 1u);
        STAMP_AT(6);
        BAR();
        if (wid == 1) {
            if (lane < NR && lane != rank) st_sys(pws + WS_SRCFLAG + rank * 2048 + 1024 + gi * 4, call);
            if (lane == 0) st_agent(ws + WS_CALL + bid * 4, call);
        }
    }
    STAMP_AT(1);

    const unsigned L = (xcd & 1) * GEMM_PER_XCD + loc, pm = L >> 2, pn = (xcd >> 1) * 4 + (L & 3);
    const unsigned wr = wid >> 2, wc = wid & 3;
    unsigned voff0, voff1;
    {
        unsigned ga = pm * 5 + wid; ga = ga > 63 ? 63 : ga;
        voff0 = (wid < 5 ? ga * GROUP_BYTES : W_OFF + (pn * 4 + wid - 5) * GROUP_BYTES) + lane * 16;
        voff1 = W_OFF + (pn * 4 + 3) * GROUP_BYTES + wid * 512 + (lane & 31) * 16;
    }
    const unsigned lds0 = (unsigned)(uintptr_t)lds;
    const unsigned a_rd = lane * 16 + wr * 10240, b_rd = lane * 16 + 20480 + wc * 4096;
    const unsigned needx = call * (pm == 12 ? 4u : 5u), needw = call * 4u;
    const char* const cntx = ws + WS_CNT + (pm * 64 + lane) * 4;
    const char* const cntw = ws + WS_CNT + ((13 + pn) * 64 + lane) * 4;
    LAS unsigned long long* const ctrl = (LAS unsigned long long*)(lds + LDS_CTRL);

    unsigned long long avail = 0, taken = 0;

    auto poll = [&](unsigned cx, unsigned cwv) {
        unsigned long long m = __ballot(cx >= needx && cwv >= needw) & ~taken;
        if (!m) {
            const unsigned long long t0 = now();
            for (unsigned it = 1;; ++it) {
                cx = ld_agent(cntx); cwv = ld_agent(cntw);
                m = __ballot(cx >= needx && cwv >= needw) & ~taken;
                if (m) break;
                if (!(it & 15) && (rfl(ld_agent(ws + WS_TMO)) == call || now() - t0 > DEADLINE)) { if (lane == 0) st_agent(ws + WS_TMO, call); m = ~taken; break; }
                __builtin_amdgcn_s_sleep(2);
            }
        }
        if (lane == 0) *ctrl = m;
        WAIT_L(0);
    };
    const unsigned voffc = (unsigned)WS_CNT + ((wid < 4 ? pm : 13 + pn) * 64 + (wid & 3) * 16 + (lane & 15)) * 4;
    auto snap = [&](unsigned which) {
        asm volatile("s_mov_b32 m0, %2\n\ts_mov_b64 exec, 0xffff\n\t"
                     "global_load_lds_dword %0, %1 sc1\n\t"
                     "s_mov_b64 exec, -1" :: "v"(voffc), "s"(ws), "s"(lds0 + LDS_SNAP + which * 512 + wid * 64) : "memory");
    };
    auto fetch_mask = [&]() {
        const unsigned long long m = *(volatile LAS unsigned long long*)ctrl;
        avail = ((unsigned long long)rfl((unsigned)(m >> 32)) << 32) | rfl((unsigned)m);
    };
    const unsigned m0w = lds0 + wid * 4096, m0h = lds0 + 32768 + wid * 512;
    auto part1 = [&](unsigned slot, unsigned long long base) {
        asm volatile("s_mov_b32 m0, %3\n\ts_nop 0\n\t"
                     "global_load_lds_dwordx4 %0, %2 sc0 sc1\n\t"
                     "global_load_lds_dwordx4 %0, %2 offset:1024 sc0 sc1\n\t"
                     "s_mov_b32 m0, %4\n\ts_mov_b32 exec_hi, 0\n\t"
                     "global_load_lds_dwordx4 %1, %2 sc0 sc1\n\t"
                     "s_mov_b32 exec_hi, -1" :: "v"(voff0), "v"(voff1), "s"(base), "s"(m0w + slot), "s"(m0h + slot) : "memory");
    };
    auto part2 = [&](unsigned slot, unsigned long long base) {
        asm volatile("s_mov_b32 m0, %2\n\ts_nop 0\n\t"
                     "global_load_lds_dwordx4 %0, %1 offset:2048 sc0 sc1\n\t"
                     "global_load_lds_dwordx4 %0, %1 offset:3072 sc0 sc1" :: "v"(voff0), "s"(base), "s"(m0w + slot) : "memory");
    };
    auto pick = [&]() {
        const unsigned t = __builtin_ctzll(avail);
        avail &= avail - 1; taken |= 1ull << t;
        return (unsigned long long)raw + t * TILE_BYTES;
    };

    f32x4 acc[10][4];
#pragma unroll
    for (int m = 0; m < 10; ++m)
#pragma unroll
        for (int n = 0; n < 4; ++n) acc[m][n] = (f32x4){0.f, 0.f, 0.f, 0.f};

    unsigned long long nb;
    bool have;
    WAIT_V(0);
    if (wid == 0) poll(0, 0);
    BAR();
    fetch_mask();
    nb = pick();
    part1(0, nb); part2(0, nb); part1(SLOT_BYTES, nb + 4096); part2(SLOT_BYTES, nb + 4096);
    have = avail != 0;
    snap(0);
    if (have) { nb = pick(); part1(2 * SLOT_BYTES, nb); WAIT_V(9); } else WAIT_V(6);
    BAR();
    if (wr == 1) BAR();
    STAMP_AT(2);

#define LOAD_B(off) _Pragma("unroll") for (int n = 0; n < 4; ++n) Bt[n] = *(const LAS f16x8*)(pb + (off) + n * 1024)
#define LOAD_A(off) _Pragma("unroll") for (int m = 0; m < 5; ++m) At[m] = *(const LAS f16x8*)(pa + (off) + m * 1024)
#define MMA(h) do { BAR(); WAIT_L(0); SCHED(); __builtin_amdgcn_s_setprio(1); \
        _Pragma("unroll") for (int m = 0; m < 5; ++m) _Pragma("unroll") for (int n = 0; n < 4; ++n) \
            acc[(h) * 5 + m][n] = __builtin_amdgcn_mfma_f32_16x16x32_f16(At[m], Bt[n], acc[(h) * 5 + m][n], 0, 0, 0); \
        __builtin_amdgcn_s_setprio(0); SCHED(); BAR(); } while (0)
    constexpr unsigned NT = 64;
    unsigned tmov = 0;
    unsigned cb = 0;
    for (unsigned i = 0; i < NT; ++i) {
        const unsigned ob = 2 * SLOT_BYTES - cb;
        const LAS char* pa = lds + cb + a_rd; const LAS char* pb = lds + cb + b_rd;
        f16x8 At[5], Bt[4];
        LOAD_B(0); LOAD_A(0);
        if (have) part2(ob, nb);
        MMA(0);
        LOAD_A(5120);
        if (have) { part1(ob + SLOT_BYTES, nb + 4096); WAIT_V(8); } else WAIT_V(0);
        MMA(1);
        const LAS unsigned* sp = (const LAS unsigned*)(lds + LDS_SNAP + (i & 1) * 512) + lane;
        const unsigned sx = sp[0], sw = sp[64];
        LOAD_B(SLOT_BYTES); LOAD_A(SLOT_BYTES);
        if (have) part2(ob + SLOT_BYTES, nb + 4096);
        else {
            avail = __ballot(sx >= needx && sw >= needw) & ~taken;
            if (avail) {
                nb = pick(); have = true;
                part1(ob, nb); part2(ob, nb); part1(ob + SLOT_BYTES, nb + 4096); part2(ob + SLOT_BYTES, nb + 4096);
            } else if (wid == 0) {
                asm volatile("s_mov_b32 m0, %2\n\ts_nop 0\n\tglobal_load_lds_dword %0, off sc1\n\t"
                             "s_mov_b32 m0, %3\n\ts_nop 0\n\tglobal_load_lds_dword %1, off sc1"
                             :: "v"(cntx), "v"(cntw), "s"(lds0 + LDS_SNAP + 1024), "s"(lds0 + LDS_SNAP + 1280) : "memory");
            }
        }
        MMA(0);
        LOAD_A(SLOT_BYTES + 5120);
        if (i + 1 < NT) {
            const bool starved = !have;
            if (starved) {
                if (wr == 0) {
                    if (wid == 0) { WAIT_V(0); const LAS unsigned* fp = (const LAS unsigned*)(lds + LDS_SNAP + 1024) + lane; poll(fp[0], fp[64]); }
                    BAR();
                }
                fetch_mask();
                nb = pick();
                part1(ob, nb); part2(ob, nb); part1(ob + SLOT_BYTES, nb + 4096); part2(ob + SLOT_BYTES, nb + 4096);
            } else avail |= __ballot(sx >= needx && sw >= needw) & ~taken;
            have = avail != 0;
            snap(~i & 1);
            if (have) { nb = pick(); part1(cb, nb); WAIT_V(9); } else WAIT_V(6);
            if (starved && wr == 1) BAR();
        } else tmov = ld_agent(ws + WS_TMO);
        MMA(1);
        cb = ob;
    }
    if (wr == 0) BAR();
    STAMP_AT(3);

    const float one = rfl(tmov) == call ? __builtin_nanf("") : 1.0f;
    const __amdgpu_buffer_rsrc_t rout = mkrsrc(P.out, (unsigned)MM * NN * 4);
    const unsigned ovo = (fq * 4 * NN + fr * 4) * 4, oso = ((pm * 320 + wr * 160) * NN + pn * 256 + wc * 64) * 4;
#pragma unroll
    for (int m = 0; m < 10; ++m) {
        if (pm * 320 + wr * 160 + m * 16 < MM) {
            const float k1 = -2.0f * 0.7978845608028654f * 1.4426950408889634f, k2 = k1 * 0.044715f;
            f32x2 y[8], z[8];
#pragma unroll
            for (int i = 0; i < 8; ++i) { y[i] = (f32x2){acc[m][i & 3][(i >> 2) * 2], acc[m][i & 3][(i >> 2) * 2 + 1]}; z[i] = y[i] * y[i]; }
            SCHED();
#pragma unroll
            for (int i = 0; i < 8; ++i) z[i] = __builtin_elementwise_fma(z[i], (f32x2){k2, k2}, (f32x2){k1, k1});
            SCHED();
#pragma unroll
            for (int i = 0; i < 8; ++i) z[i] = z[i] * y[i];
            SCHED();
#pragma unroll
            for (int i = 0; i < 8; ++i) z[i] = (f32x2){__builtin_amdgcn_exp2f(z[i].x), __builtin_amdgcn_exp2f(z[i].y)};
            SCHED();
#pragma unroll
            for (int i = 0; i < 8; ++i) z[i] = z[i] + one;
            SCHED();
#pragma unroll
            for (int i = 0; i < 8; ++i) z[i] = (f32x2){__builtin_amdgcn_rcpf(z[i].x), __builtin_amdgcn_rcpf(z[i].y)};
            SCHED();
            f32x4 o[4];
#pragma unroll
            for (int i = 0; i < 8; ++i) {
                float a0 = y[i].x * z[i].x, a1 = y[i].y * z[i].y;
                asm("" : "+v"(a0)); asm("" : "+v"(a1));
                o[(i >> 2) * 2][i & 3] = a0; o[(i >> 2) * 2 + 1][i & 3] = a1;
            }
#pragma unroll
            for (int j = 0; j < 4; ++j) __builtin_amdgcn_raw_buffer_store_b128(__builtin_bit_cast(u32x4, o[j]), rout, ovo, oso + (m * 16 + j) * NN * 4, OUT_AUX);
            SCHED();
        }
    }
}

}

extern "C" void kernel_launch(int rank, int nranks, ncclComm_t comm,
                              hipStream_t stream,
                              void* const* d_in, const int* in_sizes, int n_in,
                              void* d_out, int out_size,
                              const int* in_dtypes, int out_dtype,
                              void* const* d_out_all,
                              void* const* d_ws_all, size_t ws_bytes) {
    (void)comm; (void)in_dtypes; (void)out_dtype; (void)d_out_all;
    if (nranks != NR || n_in != 2 || (unsigned)rank >= (unsigned)NR || ws_bytes < WS_END || out_size != MM * NN ||
        in_sizes[0] != MM * KS || in_sizes[1] != KS * NN) {
        fprintf(stderr, "kernel_launch: unexpected problem shape\n");
        return;
    }
    HIP_CHECK(hipFuncSetAttribute((const void*)fused, hipFuncAttributeMaxDynamicSharedMemorySize, LDS_BYTES));
    Params P;
    P.x = (const float*)d_in[0]; P.w = (const float*)d_in[1]; P.out = (float*)d_out;
    for (int i = 0; i < NR; ++i) P.ws[i] = (char*)d_ws_all[i];
    P.self = (char*)d_ws_all[rank]; P.rank = rank; P.pad = 0;
    hipLaunchKernelGGL(fused, dim3(NWG), dim3(NTHREADS), LDS_BYTES, stream, P);
    HIP_CHECK(hipGetLastError());
}
```

```cpp
#include <hip/hip_runtime.h>
#include <rccl/rccl.h>
#include <cstdio>
#include <cstdint>
#include <type_traits>

#define HIP_CHECK(expr) do { hipError_t _e = (expr); if (_e != hipSuccess) { \
    fprintf(stderr, "HIP error %s at %s:%d\n", hipGetErrorString(_e), __FILE__, __LINE__); \
    return; } } while (0)

namespace {

typedef unsigned u32x4 __attribute__((ext_vector_type(4)));
typedef float f32x4 __attribute__((ext_vector_type(4)));
typedef float f32x2 __attribute__((ext_vector_type(2)));
typedef _Float16 f16x8 __attribute__((ext_vector_type(8)));
typedef _Float16 f16x2 __attribute__((ext_vector_type(2)));
#define LAS __attribute__((address_space(3)))
#define GAS __attribute__((address_space(1)))

constexpr int NR = 8;
constexpr int MM = 4096, NN = 4096, KS = 512;
constexpr int NWG = 256, NTHREADS = 512;
constexpr int GEMM_PER_XCD = 24;
constexpr int NGEMM = 8 * GEMM_PER_XCD, NCOMM = NWG - NGEMM;
constexpr int NGROUPS = 1024;
constexpr int GPW = 6;
constexpr int STRIP_G = 56;
constexpr int PER_TILE = 121;
constexpr int NIN = 8 * PER_TILE, NTICKET = NIN + 128;
#define PULL_NE 9
#define PULL_WIN 8
#define HEAD_NOW 64
#define HEAD_GAP 100
#define HEAD_SLOPE 10
#define OUT_AUX 18
#define RAW_HEAD 32
#define PUSH_HEAD 32
#define PUSH_WIN 12
#define DEEP_FROM 192
#define SIMPLE_TAIL 80
#define ROT_FROM 6
#define PULL_PIPE 1
#define GLDS_SC " sc0 sc1"
#define USE_TABLE 1
#define PRETOUCH 1
#define SNAP_THIN 1
constexpr unsigned long long DEADLINE = 30000000ull;

constexpr size_t WS_CALL = 0;
constexpr size_t WS_TMO = 4096;
constexpr size_t WS_PAD = 1u << 20;
constexpr size_t WS_ZERO = 3u << 19;
constexpr size_t WS_TICKET = 5120;
constexpr size_t WS_OSEQ = 6144;
constexpr size_t WS_OUTFLAG = 7168;
constexpr size_t WS_CNT = 8192;
constexpr size_t WS_SRCFLAG = 32768;
constexpr size_t WS_STAMP = 65536;
constexpr size_t WS_TRACE = 524288;
constexpr size_t WS_RAW = 2u << 20;
constexpr size_t RAW_BYTES = 66u << 20;
constexpr unsigned PACK_STRIDE = 7424;
constexpr size_t WS_PACK = WS_RAW + 2 * RAW_BYTES;
constexpr unsigned PACK_CHUNK = 1u << 20;
constexpr size_t PACK_BYTES = 8u << 20;
constexpr unsigned DEEP_STRIDE = 7680;
constexpr size_t WS_DEEP = WS_PACK + 2 * PACK_BYTES;
constexpr size_t WS_XOUT = WS_DEEP + 2 * PACK_BYTES;
constexpr size_t WS_END = WS_XOUT + (2u << 20);
constexpr unsigned TILE_BYTES = (1u << 20) + 4096  , W_OFF = 512u << 10, GROUP_BYTES = 8192;

constexpr int SLOT_BYTES = 36 * 1024;
constexpr int LDS_CTRL = 4 * SLOT_BYTES;
constexpr int LDS_SNAP = LDS_CTRL + 64;
constexpr int LDS_BYTES = LDS_SNAP + 2560;

constexpr int NDEC = 6, ENG_EX = 4 * 8192, ENG_CTL = ENG_EX + NDEC * 4096;

struct Params {
    const float* x; const float* w; float* out;
    char* self;
    char* ws[NR];
    int rank; int pad;
};

__device__ __forceinline__ unsigned rfl(unsigned v) { return __builtin_amdgcn_readfirstlane(v); }
__device__ __forceinline__ __amdgpu_buffer_rsrc_t mkrsrc(const void* p, unsigned bytes) {
    unsigned long long a = (unsigned long long)p;
    a = ((unsigned long long)rfl((unsigned)(a >> 32)) << 32) | rfl((unsigned)a);
    return __builtin_amdgcn_make_buffer_rsrc((void*)a, 0, bytes, 0x00020000);
}
__device__ __forceinline__ unsigned ld_sys(const void* p) { return __hip_atomic_load((const GAS unsigned*)p, __ATOMIC_RELAXED, __HIP_MEMORY_SCOPE_SYSTEM); }
__device__ __forceinline__ unsigned ld_agent(const void* p) { return __hip_atomic_load((const GAS unsigned*)p, __ATOMIC_RELAXED, __HIP_MEMORY_SCOPE_AGENT); }
__device__ __forceinline__ void st_sys(void* p, unsigned v) { __hip_atomic_store((GAS unsigned*)p, v, __ATOMIC_RELAXED, __HIP_MEMORY_SCOPE_SYSTEM); }
__device__ __forceinline__ void st_agent(void* p, unsigned v) { __hip_atomic_store((GAS unsigned*)p, v, __ATOMIC_RELAXED, __HIP_MEMORY_SCOPE_AGENT); }
__device__ __forceinline__ void add_agent(void* p, unsigned v) {
    unsigned z = 0; asm("" : "+v"(z));
    __hip_atomic_fetch_add((GAS unsigned*)p + z, v, __ATOMIC_RELAXED, __HIP_MEMORY_SCOPE_AGENT);
}
__device__ __forceinline__ unsigned lds0_of(LAS char* p) { return (unsigned)(uintptr_t)p; }
__device__ __forceinline__ unsigned long long now() { return __builtin_amdgcn_s_memrealtime(); }
#define WAIT_V(n) asm volatile("s_waitcnt vmcnt(" #n ")" ::: "memory")
#define WAIT_V0() do { __builtin_amdgcn_s_waitcnt(0x0F70); asm volatile("s_waitcnt vmcnt(0)" ::: "memory"); } while (0)
#define WAIT_L(n) asm volatile("s_waitcnt lgkmcnt(" #n ")" ::: "memory")
#define BAR() do { asm volatile("" ::: "memory"); __builtin_amdgcn_s_barrier(); asm volatile("" ::: "memory"); } while (0)
#define SCHED() __builtin_amdgcn_sched_barrier(0)
#define KEEP1(a) asm volatile("s_nop 1" :: "v"(a) : "memory")
#define KEEP4(a) asm volatile("s_nop 1" :: "v"((a)[0]), "v"((a)[1]), "v"((a)[2]), "v"((a)[3]) : "memory")
#define KEEP8(a) asm volatile("s_nop 1" :: "v"((a)[0]), "v"((a)[1]), "v"((a)[2]), "v"((a)[3]), "v"((a)[4]), "v"((a)[5]), "v"((a)[6]), "v"((a)[7]) : "memory")

#define STAMP_AT(slot) do {} while (0)

__device__ __forceinline__ void order(unsigned q, unsigned& c, unsigned& isw, unsigned& g) { c = q >> 7; isw = q & 1; g = (q & 127) >> 1; }
__device__ const __attribute__((aligned(4))) unsigned char LAST_ORDER[7][256] = {
    {201,203,205,207,209,211,213,215,145,147,149,151,240,238,241,243,245,247,137,139,141,143,148,150,152,154,156,178,180,182,184,186,185,187,189,191,225,227,229,231,228,230,232,234,236,217,219,221,223,208,210,212,214,216,168,170,172,174,176,128,130,132,134,136,188,190,192,194,196,218,220,222,224,226,161,163,165,167,112,110,138,140,142,144,146,81,83,85,87,198,200,202,204,206,129,131,133,135,249,251,253,255,113,115,117,119,57,59,61,63,97,99,101,103,169,171,173,175,158,160,162,164,166,80,82,84,86,88,153,155,157,159,89,91,93,95,177,179,181,183,1,3,5,7,40,42,44,46,48,233,235,237,239,193,195,197,199,49,51,53,55,33,35,37,39,17,19,21,23,105,107,109,111,121,123,125,127,60,62,64,66,68,25,27,29,31,50,52,54,56,58,100,102,104,106,108,73,75,77,79,65,67,69,71,30,32,34,36,38,9,11,13,15,20,22,24,26,28,41,43,45,47,10,12,14,16,18,90,92,94,96,98,70,72,74,76,78,0,2,4,6,8},
    {240,238,40,42,44,46,48,137,139,141,143,57,59,61,63,65,67,69,71,241,243,245,247,112,110,113,115,117,119,209,211,213,215,90,92,94,96,98,81,83,85,87,89,91,93,95,70,72,74,76,78,225,227,229,231,158,160,162,164,166,25,27,29,31,33,35,37,39,50,52,54,56,58,10,12,14,16,18,0,2,4,6,8,97,99,101,103,80,82,84,86,88,1,3,5,7,128,130,132,134,136,17,19,21,23,161,163,165,167,100,102,104,106,108,20,22,24,26,28,49,51,53,55,105,107,109,111,73,75,77,79,30,32,34,36,38,60,62,64,66,68,41,43,45,47,121,123,125,127,145,147,149,151,217,219,221,223,178,180,182,184,186,198,200,202,204,206,138,140,142,144,146,129,131,133,135,169,171,173,175,185,187,189,191,9,11,13,15,208,210,212,214,216,168,170,172,174,176,177,179,181,183,228,230,232,234,236,153,155,157,159,249,251,253,255,218,220,222,224,226,148,150,152,154,156,233,235,237,239,193,195,197,199,201,203,205,207,188,190,192,194,196},
    {161,163,165,167,240,238,241,243,245,247,33,35,37,39,208,210,212,214,216,129,131,133,135,178,180,182,184,186,168,170,172,174,176,209,211,213,215,217,219,221,223,148,150,152,154,156,112,110,228,230,232,234,236,158,160,162,164,166,201,203,205,207,89,91,93,95,41,43,45,47,138,140,142,144,146,185,187,189,191,113,115,117,119,145,147,149,151,198,200,202,204,206,188,190,192,194,196,225,227,229,231,193,195,197,199,50,52,54,56,58,17,19,21,23,97,99,101,103,128,130,132,134,136,137,139,141,143,81,83,85,87,218,220,222,224,226,249,251,253,255,40,42,44,46,48,10,12,14,16,18,57,59,61,63,153,155,157,159,1,3,5,7,90,92,94,96,98,121,123,125,127,25,27,29,31,9,11,13,15,80,82,84,86,88,100,102,104,106,108,49,51,53,55,233,235,237,239,0,2,4,6,8,169,171,173,175,70,72,74,76,78,20,22,24,26,28,177,179,181,183,30,32,34,36,38,73,75,77,79,60,62,64,66,68,65,67,69,71,105,107,109,111},
    {168,170,172,174,176,138,140,142,144,146,17,19,21,23,225,227,229,231,112,110,137,139,141,143,145,147,149,151,240,238,217,219,221,223,188,190,192,194,196,228,230,232,234,236,198,200,202,204,206,158,160,162,164,166,81,83,85,87,129,131,133,135,153,155,157,159,113,115,117,119,209,211,213,215,241,243,245,247,233,235,237,239,1,3,5,7,177,179,181,183,89,91,93,95,208,210,212,214,216,185,187,189,191,161,163,165,167,169,171,173,175,249,251,253,255,10,12,14,16,18,97,99,101,103,40,42,44,46,48,178,180,182,184,186,33,35,37,39,148,150,152,154,156,193,195,197,199,218,220,222,224,226,128,130,132,134,136,57,59,61,63,50,52,54,56,58,70,72,74,76,78,201,203,205,207,30,32,34,36,38,9,11,13,15,80,82,84,86,88,41,43,45,47,60,62,64,66,68,90,92,94,96,98,65,67,69,71,0,2,4,6,8,25,27,29,31,121,123,125,127,49,51,53,55,105,107,109,111,100,102,104,106,108,73,75,77,79,20,22,24,26,28},
    {178,180,182,184,186,241,243,245,247,218,220,222,224,226,225,227,229,231,128,130,132,134,136,112,110,198,200,202,204,206,129,131,133,135,158,160,162,164,166,138,140,142,144,146,57,59,61,63,177,179,181,183,240,238,168,170,172,174,176,145,147,149,151,161,163,165,167,137,139,141,143,185,187,189,191,113,115,117,119,217,219,221,223,33,35,37,39,188,190,192,194,196,17,19,21,23,30,32,34,36,38,81,83,85,87,148,150,152,154,156,89,91,93,95,209,211,213,215,40,42,44,46,48,249,251,253,255,1,3,5,7,153,155,157,159,169,171,173,175,228,230,232,234,236,0,2,4,6,8,9,11,13,15,208,210,212,214,216,233,235,237,239,97,99,101,103,100,102,104,106,108,201,203,205,207,193,195,197,199,10,12,14,16,18,20,22,24,26,28,25,27,29,31,90,92,94,96,98,70,72,74,76,78,121,123,125,127,41,43,45,47,73,75,77,79,49,51,53,55,60,62,64,66,68,105,107,109,111,50,52,54,56,58,80,82,84,86,88,65,67,69,71},
    {129,131,133,135,161,163,165,167,138,140,142,144,146,217,219,221,223,225,227,229,231,218,220,222,224,226,185,187,189,191,178,180,182,184,186,112,110,241,243,245,247,240,238,89,91,93,95,41,43,45,47,128,130,132,134,136,97,99,101,103,198,200,202,204,206,145,147,149,151,168,170,172,174,176,153,155,157,159,158,160,162,164,166,49,51,53,55,228,230,232,234,236,209,211,213,215,148,150,152,154,156,100,102,104,106,108,137,139,141,143,233,235,237,239,193,195,197,199,70,72,74,76,78,113,115,117,119,81,83,85,87,33,35,37,39,25,27,29,31,9,11,13,15,188,190,192,194,196,208,210,212,214,216,201,203,205,207,177,179,181,183,169,171,173,175,0,2,4,6,8,249,251,253,255,20,22,24,26,28,17,19,21,23,60,62,64,66,68,57,59,61,63,65,67,69,71,90,92,94,96,98,1,3,5,7,105,107,109,111,73,75,77,79,10,12,14,16,18,80,82,84,86,88,50,52,54,56,58,40,42,44,46,48,30,32,34,36,38,121,123,125,127},
    {240,238,112,110,158,160,162,164,166,89,91,93,95,9,11,13,15,113,115,117,119,185,187,189,191,33,35,37,39,209,211,213,215,129,131,133,135,145,147,149,151,208,210,212,214,216,198,200,202,204,206,153,155,157,159,128,130,132,134,136,81,83,85,87,217,219,221,223,218,220,222,224,226,137,139,141,143,148,150,152,154,156,241,243,245,247,228,230,232,234,236,225,227,229,231,188,190,192,194,196,233,235,237,239,161,163,165,167,25,27,29,31,169,171,173,175,201,203,205,207,193,195,197,199,178,180,182,184,186,177,179,181,183,138,140,142,144,146,57,59,61,63,90,92,94,96,98,249,251,253,255,70,72,74,76,78,17,19,21,23,20,22,24,26,28,49,51,53,55,97,99,101,103,30,32,34,36,38,168,170,172,174,176,41,43,45,47,1,3,5,7,105,107,109,111,10,12,14,16,18,0,2,4,6,8,40,42,44,46,48,50,52,54,56,58,73,75,77,79,100,102,104,106,108,80,82,84,86,88,121,123,125,127,65,67,69,71,60,62,64,66,68},
};
__device__ __forceinline__ unsigned pack_off(unsigned q) { return (q >> 7) * PACK_CHUNK + (q & 127u) * PACK_STRIDE; }
__device__ __forceinline__ unsigned deep_off(unsigned q) { return (q >> 7) * PACK_CHUNK + (q & 127u) * DEEP_STRIDE; }
__device__ __forceinline__ unsigned group_off(unsigned c, unsigned isw, unsigned g, unsigned sender) { return (c * 8 + sender) * TILE_BYTES + isw * W_OFF + g * GROUP_BYTES; }
__device__ __forceinline__ unsigned cnt_idx(unsigned c, unsigned isw, unsigned g, unsigned sender) {
    const unsigned panel = isw ? 13u + (g >> 2) : g >= 55u ? 11u : g / 5u;
    return panel * 64u + c * 8u + sender;
}
__device__ __forceinline__ unsigned pk(float a, float b) { f32x2 v = {a, b}; return __builtin_bit_cast(unsigned, __builtin_convertvector(v, f16x2)); }

typedef unsigned short u16x2 __attribute__((ext_vector_type(2)));
__device__ __forceinline__ unsigned bfi(unsigned m, unsigned a, unsigned b) { return (a & m) | (b & ~m); }
__device__ __forceinline__ unsigned pk_add16(unsigned a, unsigned b) { return __builtin_bit_cast(unsigned, __builtin_bit_cast(u16x2, a) + __builtin_bit_cast(u16x2, b)); }
__device__ __forceinline__ unsigned pk_sub16(unsigned a, unsigned b) { return __builtin_bit_cast(unsigned, __builtin_bit_cast(u16x2, a) - __builtin_bit_cast(u16x2, b)); }
__device__ __forceinline__ unsigned pk_min16(unsigned a, unsigned b) { return __builtin_bit_cast(unsigned, __builtin_elementwise_min(__builtin_bit_cast(u16x2, a), __builtin_bit_cast(u16x2, b))); }
__device__ __forceinline__ unsigned pk_max16(unsigned a, unsigned b) { return __builtin_bit_cast(unsigned, __builtin_elementwise_max(__builtin_bit_cast(u16x2, a), __builtin_bit_cast(u16x2, b))); }
__device__ __forceinline__ unsigned wave_scan(unsigned x) {
    x += __builtin_amdgcn_update_dpp(0u, x, 0x111, 0xf, 0xf, true); x += __builtin_amdgcn_update_dpp(0u, x, 0x112, 0xf, 0xf, true);
    x += __builtin_amdgcn_update_dpp(0u, x, 0x114, 0xf, 0xf, true); x += __builtin_amdgcn_update_dpp(0u, x, 0x118, 0xf, 0xf, true);
    x += __builtin_amdgcn_update_dpp(0u, x, 0x142, 0xa, 0xf, false); x += __builtin_amdgcn_update_dpp(0u, x, 0x143, 0xc, 0xf, false);
    return x;
}
#define LDS_FENCE() do { asm volatile("" ::: "memory"); __builtin_amdgcn_wave_barrier(); } while (0)

__global__ void __launch_bounds__(NTHREADS, 2) fused(Params P) {
    extern __shared__ __attribute__((aligned(16))) char smem[];
    LAS char* lds = (LAS char*)smem;
    const unsigned tid = threadIdx.x, lane = tid & 63, wid = rfl(tid >> 6);
    const unsigned bid = blockIdx.x, xcd = bid & 7, loc = bid >> 3;
    unsigned long long a_x = (unsigned long long)P.x, a_w = (unsigned long long)P.w, a_out = (unsigned long long)P.out, a_self = (unsigned long long)P.self;
    unsigned rank = (unsigned)P.rank;
    int wlo = 0, whi = 0;
#define PUT_WS(i) asm("v_writelane_b32 %0, %2, " #i "\n\tv_writelane_b32 %1, %3, " #i : "+v"(wlo), "+v"(whi) : "s"((unsigned)(unsigned long long)P.ws[i]), "s"((unsigned)((unsigned long long)P.ws[i] >> 32)))
    PUT_WS(0); PUT_WS(1); PUT_WS(2); PUT_WS(3); PUT_WS(4); PUT_WS(5); PUT_WS(6); PUT_WS(7);
    asm volatile("" : "+s"(a_x), "+s"(a_w), "+s"(a_out), "+s"(a_self), "+s"(rank));
    asm volatile("s_nop 1" : "+v"(wlo), "+v"(whi));
    char* const ws = (char*)a_self;
    STAMP_AT(0);
    char* const callp = ws + WS_CALL + bid * 4;
    const unsigned callv = ld_agent(callp);

    if (loc >= GEMM_PER_XCD) {
        const unsigned role = wid;
        LAS char* const eng = lds;
        volatile LAS unsigned* const ctl = (volatile LAS unsigned*)(eng + ENG_CTL);
        if (role == 0 && lane < 32) ctl[lane] = 0;
        const unsigned call = rfl(callv) + 1u, parity = call & 1u;
        WAIT_V(0); WAIT_L(0); BAR();
        if (tid == 0) st_agent(callp, call);
        const unsigned e = (loc - GEMM_PER_XCD) * 8 + xcd;
        if (e >= 7u * PULL_NE) return;
        const unsigned link = e % 7u, peer = (rank + 1u + link) & 7u, idx = e / 7u;
        const unsigned long long pbase = ((unsigned long long)(unsigned)__builtin_amdgcn_readlane(whi, peer) << 32) | (unsigned)__builtin_amdgcn_readlane(wlo, peer);
        const unsigned raw_so = (unsigned)WS_RAW + parity * (unsigned)RAW_BYTES, pak_so = (unsigned)WS_PACK + parity * (unsigned)PACK_BYTES;
        const unsigned out_so = (unsigned)WS_XOUT + parity * (1u << 20), deep_so = (unsigned)WS_DEEP + parity * (unsigned)PACK_BYTES;
        auto goff = [&](unsigned q) { unsigned c, isw, g; order(q, c, isw, g); return group_off(c, isw, g, peer); };
        constexpr unsigned DEAD = 0x10000u, STOP = 0xffffu, DEEP = 0x1000u;

        if (role == 1) {
            const char* sflag = ws + WS_SRCFLAG + peer * 2048;
            const unsigned long long t0 = now();
            unsigned known = 0;
            for (unsigned it = 1; known < NGROUPS; ++it) {
                unsigned pre;
                if (known < 64) {
                    const unsigned long long miss = __builtin_amdgcn_ballot_w64(ld_sys(sflag + lane * 4) < call && lane >= PUSH_HEAD);
                    pre = miss ? __builtin_ctzll(miss) : 64u;
                } else {
                    unsigned v[8]; bool all1 = true;
#pragma unroll
                    for (int k = 1; k < 8; ++k) v[k] = ld_sys(sflag + (k * 64 + lane) * 4);
                    pre = NGEMM;
#pragma unroll
                    for (int k = 7; k >= 1; --k) {
                        const bool ok = v[k] >= call || ((k * 64 + lane) & 255u) >= NGEMM;
                        if (k < 4) { const unsigned long long miss = __builtin_amdgcn_ballot_w64(!ok); if (miss) pre = k * 64 + __builtin_ctzll(miss); } else all1 &= ok;
                    }
                    if (pre == NGEMM && __all(all1)) pre = NGROUPS;
                }
                if (!(it & 15) && (rfl(ld_agent(ws + WS_TMO)) == call || now() - t0 > DEADLINE)) { if (lane == 0) st_agent(ws + WS_TMO, call); pre = DEAD; }
                if (pre != known) { known = pre; ctl[8] = known; }
                if (known < 64) __builtin_amdgcn_s_sleep(1); else __builtin_amdgcn_s_sleep(24);
            }
            if (idx == 0 && known == NGROUPS && lane < PUSH_HEAD) add_agent(ws + WS_CNT + parity * 8192 + cnt_idx(0, lane & 1, lane >> 1, peer) * 4, 1u);
            for (unsigned it = 1; known != DEAD; ++it) {
                const unsigned v = ld_sys(ws + WS_OUTFLAG + peer * 128 + (lane & 15) * 4);
                if (lane < 16) ctl[16 + lane] = v;
                if (__all((v >> 4) >= call)) break;
                if (!(it & 15) && (rfl(ld_agent(ws + WS_TMO)) == call || now() - t0 > DEADLINE)) { if (lane == 0) st_agent(ws + WS_TMO, call); known = DEAD; ctl[8] = DEAD; }
                if (rfl(ctl[9])) __builtin_amdgcn_s_sleep(2); else __builtin_amdgcn_s_sleep(64);
            }
            if (known == DEAD && lane == 0) ((float*)a_out)[(size_t)(STRIP_G * 64 + peer * 64) * NN] = __builtin_nanf("");
            return;
        }

        if (role == 0) {
            const unsigned ordv = *(const GAS unsigned*)(&LAST_ORDER[link][0] + lane * 4);
            const unsigned tbias = PUSH_HEAD + PULL_NE - (call - 1u) * (NTICKET - PUSH_HEAD);
            const unsigned vo = lane * 16, tko = (unsigned)WS_TICKET + peer * 128;
            const unsigned m0e = lds0_of(eng);
            unsigned q = PUSH_HEAD + idx, n = 0, post = 0, known = 0, gate = 0, freev = 0;
            auto open = [&]() { return q < NGROUPS ? q < known : (gate >> 4) >= call; };
            unsigned one = 1, deep = 0;
            __builtin_amdgcn_s_setprio(3);
            WAIT_V0();
            for (;; ++n) {
                const unsigned slot = n & 3;
                if (!open() || freev < (n >> 2)) {
                    WAIT_V(0);
                    if (n) ctl[(n - 1) & 3] = post;
                    for (;;) {
                        known = rfl(ctl[8]); gate = rfl(ctl[16 + ((q >> 3) & 15)]); freev = rfl(ctl[4 + slot]);
                        if ((open() && freev >= (n >> 2)) || known == DEAD) break;
                        __builtin_amdgcn_s_sleep(1);
                    }
                    if (known == DEAD) break;
                }
                if (q >= NGROUPS) q = NGROUPS + (gate & 15u) * 8 + (q & 7u);
                const bool isp = q >= RAW_HEAD && q < NGROUPS;
                const unsigned so = deep ? deep_so + deep_off(q) : isp ? pak_so + pack_off(q) : q < NGROUPS ? raw_so + goff(q) : out_so + (q - NGROUPS) * 8192;
                const unsigned long long sb = pbase + rfl(so), sb2 = sb + 4096;
                const unsigned long long ex7 = deep ? 0xffffffffull : ~0ull, ex8 = deep ? 0xffffffull : rfl(isp) ? 0xffull : ~0ull;
                const unsigned m0a = m0e + slot * 8192;
                unsigned tk, tks = 0, kv = 0, fv = 0, gv = 0;
#define L_STEP(k, text, ...) do { \
                    asm volatile("s_waitcnt vmcnt(%0)" :: "i"(PULL_WIN - 1) : "memory"); \
                    if ((k) == PULL_WIN - 1) { if (n) ctl[(n - 1) & 3] = post; } \
                    if ((k) == PULL_WIN) asm volatile("v_readfirstlane_b32 %0, %1" : "=s"(tks) : "v"(tk)); \
                    if ((k) == 7) { kv = ctl[8]; fv = ctl[4 + ((n + 1) & 3)]; gv = ctl[16 + (lane & 15)]; } \
                    asm volatile(text :: __VA_ARGS__ : "memory"); } while (0)
                asm volatile("s_waitcnt vmcnt(%3)\n\ts_mov_b64 exec, 1\n\tglobal_atomic_add %0, %1, %2, %4 sc0\n\ts_mov_b64 exec, -1"
                             : "=&v"(tk) : "v"(tko), "v"(one), "i"(PULL_WIN - 1), "s"(ws) : "memory");
                L_STEP(1, "s_mov_b32 m0, %2\n\ts_nop 0\n\tglobal_load_lds_dwordx4 %0, %1 sc0 sc1", "v"(vo), "s"(sb), "s"(m0a));
                L_STEP(2, "global_load_lds_dwordx4 %0, %1 offset:1024 sc0 sc1", "v"(vo), "s"(sb));
                L_STEP(3, "global_load_lds_dwordx4 %0, %1 offset:2048 sc0 sc1", "v"(vo), "s"(sb));
                L_STEP(4, "global_load_lds_dwordx4 %0, %1 offset:3072 sc0 sc1", "v"(vo), "s"(sb));
                L_STEP(5, "s_mov_b32 m0, %2\n\ts_nop 0\n\tglobal_load_lds_dwordx4 %0, %1 sc0 sc1", "v"(vo), "s"(sb2), "s"(m0a + 4096));
                L_STEP(6, "global_load_lds_dwordx4 %0, %1 offset:1024 sc0 sc1", "v"(vo), "s"(sb2));
                L_STEP(7, "s_mov_b64 exec, %2\n\tglobal_load_lds_dwordx4 %0, %1 offset:2048 sc0 sc1\n\ts_mov_b64 exec, -1", "v"(vo), "s"(sb2), "s"(ex7));
                L_STEP(8, "s_mov_b64 exec, %2\n\tglobal_load_lds_dwordx4 %0, %1 offset:3072 sc0 sc1\n\ts_mov_b64 exec, -1", "v"(vo), "s"(sb2), "s"(ex8));
                if (PULL_WIN == 9) asm volatile("s_waitcnt vmcnt(8)\n\tv_readfirstlane_b32 %0, %1" : "=s"(tks) : "v"(tk) : "memory");
                post = (((n >> 2) + 1) << 16) | deep | q;
                const unsigned tq = tks + tbias;
                if (tq >= NTICKET) { WAIT_V(0); ctl[slot] = post; ++n; break; }
                if (tq < NIN) {
                    unsigned c = tq / PER_TILE, j = tq - c * PER_TILE;
                    if (c >= 2) { j += 110; j = j >= PER_TILE ? j - PER_TILE : j; }
                    unsigned r = j <= 112u ? j : 2 * j - 113u;
                    if (c >= 6) {
                        const unsigned k = tq - 6 * PER_TILE, ent = (__builtin_amdgcn_readlane(ordv, k >> 2) >> ((k & 3u) * 8u)) & 255u;
                        c = 6 + (ent >> 7); r = ent & 127u;
                    }
                    q = c * 128 + (r == 112u ? 112u + 2 * rank : r);
                    if (tq >= 7 * PER_TILE) ctl[9] = 1;
                } else q = NGROUPS + tq - NIN;
                deep = q >= DEEP_FROM && tq < NIN - SIMPLE_TAIL ? DEEP : 0u;
                gate = __builtin_amdgcn_readlane(gv, (q >> 3) & 15);
                known = rfl(kv); freev = rfl(fv);
                if (known == DEAD) { WAIT_V(0); ctl[slot] = post; ++n; break; }
            }
            for (unsigned k = 0; k < NDEC; ++k, ++n) {
                while (rfl(ctl[4 + (n & 3)]) < (n >> 2)) __builtin_amdgcn_s_sleep(1);
                ctl[n & 3] = (((n >> 2) + 1) << 16) | STOP;
            }
            return;
        }

        const __amdgpu_buffer_rsrc_t rsrc = __builtin_amdgcn_make_buffer_rsrc((void*)pbase, 0, (unsigned)WS_END, 0x00020000);
        const __amdgpu_buffer_rsrc_t rdst = mkrsrc(ws + WS_RAW + parity * RAW_BYTES, (unsigned)RAW_BYTES);
        const __amdgpu_buffer_rsrc_t rout = mkrsrc((void*)a_out, (unsigned)MM * NN * 4);
        LAS unsigned* const ex = (LAS unsigned*)(eng + ENG_EX + (role - 2) * 4096);
        for (unsigned n = role - 2;; n += NDEC) {
            const unsigned slot = n & 3;
#pragma unroll
            for (int a = 0; a < 5; ++a) ex[a * 64 + lane] = 0;
            unsigned w;
            while (((w = rfl(ctl[slot])) >> 16) <= (n >> 2)) __builtin_amdgcn_s_sleep(1);
            if ((w & 0xffffu) == STOP) { ctl[4 + slot] = (n >> 2) + 1; break; }
            const unsigned q = w & 0xfffu;
            asm volatile("" ::: "memory");
            const LAS char* sp = eng + slot * 8192;
            if (w & DEEP) {
                const unsigned len = *(const LAS unsigned char*)(sp + 7168 + lane);
                const unsigned hdr = rfl(*(const LAS unsigned*)(sp + 7232)), mode = hdr >> 16;
                unsigned D[32];
#pragma unroll
                for (int v = 0; v < 6; ++v) { const u32x4 t = *(const LAS u32x4*)(sp + v * 1024 + lane * 16); D[4 * v] = t[0]; D[4 * v + 1] = t[1]; D[4 * v + 2] = t[2]; D[4 * v + 3] = t[3]; }
                { const unsigned long long t = *(const LAS unsigned long long*)(sp + 6144 + lane * 8); D[24] = (unsigned)t; D[25] = (unsigned)(t >> 32); }
                if (mode == 1) {
                    const unsigned x = __builtin_amdgcn_raw_buffer_load_b32(rsrc, 7552 + (lane & 31) * 4, deep_so + deep_off(q), 17);
                    *(LAS unsigned*)(sp + 7552 + (lane & 31) * 4) = x;
                    LDS_FENCE();
                }
                const unsigned off = (wave_scan(len) - len) & 0xfffu;
                const LAS unsigned* bp = (const LAS unsigned*)(sp + 7236) + (off >> 5);
                unsigned b0 = bp[0], b1 = bp[1], b2w = bp[2], b3 = bp[3], b4 = bp[4];
#pragma unroll
                for (int i = 0; i < 26; ++i) asm volatile("" : "+v"(D[i]));
                asm volatile("" : "+v"(b0), "+v"(b1), "+v"(b2w), "+v"(b3), "+v"(b4));
                WAIT_L(0);
                ctl[4 + slot] = (n >> 2) + 1;
                const unsigned so = goff(q);
                if (mode > 1) {
                    u32x4 R[8];
#pragma unroll
                    for (int v = 0; v < 8; ++v) R[v] = __builtin_amdgcn_raw_buffer_load_b128(rsrc, lane * 16 + v * 1024, raw_so + so, 17);
#pragma unroll
                    for (int v = 0; v < 8; ++v) __builtin_amdgcn_raw_buffer_store_b128(R[v], rdst, lane * 16 + v * 1024, so, 17);
                    KEEP8(R);
                } else {
#pragma unroll
                    for (int j = 0; j < 6; ++j) {
                        unsigned h = D[24 + j / 3] << (3 - j % 3);
#pragma unroll
                        for (int k = 0; k < 4; ++k) { h = bfi(0x00070007u << (3 * k), D[4 * j + k] >> (12 - 3 * k), h); asm("" : "+v"(h)); }
                        D[26 + j] = h;
                    }
                    const unsigned a0 = __builtin_amdgcn_alignbit(b1, b0, off), a1 = __builtin_amdgcn_alignbit(b2w, b1, off),
                                   a2 = __builtin_amdgcn_alignbit(b3, b2w, off), a3 = __builtin_amdgcn_alignbit(b4, b3, off);
                    const unsigned base2 = (hdr & 0xffffu) * 0x00010001u;
                    unsigned lo = a0, hi = a1, used = 0;
#pragma unroll
                    for (int v = 0; v < 8; ++v) {
                        if (v == 4) {
                            const unsigned nb = (used & 0xffffu) + (used >> 16);
                            const unsigned x0 = nb < 32 ? a0 : nb < 64 ? a1 : a2, x1 = nb < 32 ? a1 : nb < 64 ? a2 : a3, x2 = nb < 32 ? a2 : a3;
                            lo = __builtin_amdgcn_alignbit(x1, x0, nb); hi = __builtin_amdgcn_alignbit(x2, x1, nb);
                        }
                        u32x4 o;
#pragma unroll
                        for (int k = 0; k < 4; ++k) {
                            const unsigned d = D[4 * v + k], e = d & (d >> 1);
                            const unsigned l0 = (__builtin_ctz(lo | 0x80000000u) + 1u) & (unsigned)__builtin_amdgcn_sbfe((int)e, 10, 1);
                            lo = __builtin_amdgcn_alignbit(hi, lo, l0); hi >>= l0 & 31;
                            const unsigned l1 = (__builtin_ctz(lo | 0x80000000u) + 1u) & (unsigned)__builtin_amdgcn_sbfe((int)e, 26, 1);
                            lo = __builtin_amdgcn_alignbit(hi, lo, l1); hi >>= l1 & 31;
                            const unsigned lp = l0 | (l1 << 16);
                            if (v < 4) used += lp;
                            const unsigned idx2 = ((d >> 10) & 0x00030003u) + pk_min16(lp, 0x00040004u);
                            const unsigned tb = __builtin_amdgcn_perm(0x01060000u, 0x00020100u, idx2);
                            o[k] = pk_add16(bfi(0x7c007c00u, pk_sub16(tb, lp) << 10, d), base2);
                        }
                        __builtin_amdgcn_raw_buffer_store_b128(o, rdst, lane * 16 + v * 1024, so, 17);
                        KEEP1(o);
                    }
                }
                WAIT_V0();
                unsigned c, isw, g; order(q, c, isw, g);
                if (lane == 0) add_agent(ws + WS_CNT + parity * 8192 + cnt_idx(c, isw, g, peer) * 4, 1u);
                continue;
            }
            unsigned rec = *(const LAS unsigned short*)(sp + 7168 + lane * 2);
            u32x4 R[8];
#pragma unroll
            for (int v = 0; v < 8; ++v) R[v] = *(const LAS u32x4*)(sp + v * 1024 + lane * 16);
            const unsigned hdr = rfl(rec), mode = q < RAW_HEAD || q >= NGROUPS ? 3u : hdr >> 12;
            if (mode < 2) {
                rec = lane ? rec : 0u;
                __hip_atomic_fetch_or(ex + (rec >> 6), (rec & 3u) << ((rec >> 1) & 30u), __ATOMIC_RELAXED, __HIP_MEMORY_SCOPE_WORKGROUP);
            }
#pragma unroll
            for (int v = 0; v < 8; ++v) asm volatile("" : "+v"(R[v]));
            WAIT_L(0);
            ctl[4 + slot] = (n >> 2) + 1;
            if (q >= NGROUPS) {
                const unsigned oso = ((STRIP_G * 64 + peer * 64 + (q & 7u) * 8) * NN + ((q >> 3) & 15u) * 256) * 4;
#pragma unroll
                for (int v = 0; v < 8; ++v) __builtin_amdgcn_raw_buffer_store_b128(R[v], rout, lane * 16, oso + v * NN * 4, OUT_AUX);
                KEEP8(R);
                continue;
            }
            const unsigned so = goff(q);
            if (mode < 2) {
                if (mode == 1) {
                    rec = __builtin_amdgcn_raw_buffer_load_b16(rsrc, 7296 + lane * 2, pak_so + pack_off(q), 17) & 0xffffu;
                    __hip_atomic_fetch_or(ex + (rec >> 6), (rec & 3u) << ((rec >> 1) & 30u), __ATOMIC_RELAXED, __HIP_MEMORY_SCOPE_WORKGROUP);
                }
                LDS_FENCE();
                unsigned E[5];
#pragma unroll
                for (int a = 0; a < 5; ++a) E[a] = ex[a * 64 + lane];
                LDS_FENCE();
                const unsigned b2 = (hdr & 0x1ffu) * 0x00400040u;
                u32x4 c7;
#pragma unroll
                for (int k = 0; k < 4; ++k) {
                    c7[k] = R[0][k] >> 13;
#pragma unroll
                    for (int v = 1; v < 7; ++v) { c7[k] = bfi(0x00030003u << (2 * v), R[v][k] >> (13 - 2 * v), c7[k]); asm("" : "+v"(c7[k])); }
                    c7[k] = bfi(0x1fff1fffu, c7[k], c7[k] << 2); asm("" : "+v"(c7[k]));
                }
                SCHED();
#pragma unroll
                for (int v = 0; v < 8; ++v) {
                    u32x4 o;
#pragma unroll
                    for (int k = 0; k < 4; ++k) o[k] = pk_add16(v < 7 ? bfi(0x60006000u, E[k] << (13 - 2 * v), R[v][k]) : bfi(0x60006000u, E[4] << (13 - 2 * k), c7[k]), b2);
                    __builtin_amdgcn_raw_buffer_store_b128(o, rdst, lane * 16 + v * 1024, so, 17);
                    KEEP1(o);
                    SCHED();
                }
            } else {
                if (mode == 2) {
#pragma unroll
                    for (int v = 0; v < 8; ++v) R[v] = __builtin_amdgcn_raw_buffer_load_b128(rsrc, lane * 16 + v * 1024, raw_so + so, 17);
                }
#pragma unroll
                for (int v = 0; v < 8; ++v) __builtin_amdgcn_raw_buffer_store_b128(R[v], rdst, lane * 16 + v * 1024, so, 17);
                KEEP8(R);
            }
            WAIT_V0();
            unsigned c, isw, g; order(q, c, isw, g);
            if (lane == 0) add_agent(ws + WS_CNT + parity * 8192 + cnt_idx(c, isw, g, peer) * 4, 1u);
        }
        STAMP_AT(2);
        return;
    }

    const unsigned fr = lane & 15, fq = lane >> 4;
    unsigned call;
    char* raw;
    {
        const unsigned gi = loc * 8 + xcd;
        const unsigned q = wid * NGEMM + gi;
        if (q >= HEAD_NOW && q < NGROUPS) {
            const unsigned long long t0 = now(), dt = HEAD_GAP + (q * HEAD_SLOPE) / 10;
            while (now() - t0 < dt) __builtin_amdgcn_s_sleep(2);
        }
        if (wid == 7 && gi < 64) {
            __builtin_amdgcn_s_sleep(10);
            char* const pw = (char*)(((unsigned long long)(unsigned)__shfl(whi, lane & 7) << 32) | (unsigned)__shfl(wlo, lane & 7));
            const unsigned k = lane >> 3;
            if (k == 4 || (k < 4 && gi < PUSH_HEAD)) {
                const unsigned dummy = ld_sys(k == 4 ? pw + WS_PAD + (rank * 64 + gi) * 128
                                                     : pw + WS_RAW + (k & 1) * RAW_BYTES + rank * TILE_BYTES + (gi & 15) * 128 + ((k & 2) ? (1u << 20) : 0u) - ((k & 2) ? 0u : 2048u));
                asm volatile("" :: "v"(dummy));
            }
        }
        unsigned c = 0, isw = 0, g = 0;
        const bool have = wid < GPW && q < NGROUPS;
        u32x4 V[8];
        float asum = 0.f;
        if (have) {
            order(q, c, isw, g);
            f32x4 L[16];
            if (!isw) {
                const __amdgpu_buffer_rsrc_t rx = mkrsrc((void*)a_x, MM * KS * 4);
                const unsigned vo = (fr * KS + fq * 4) * 4, so = (g * 64 * KS + c * 64) * 4;
#pragma unroll
                for (int b = 0; b < 4; ++b)
#pragma unroll
                    for (int u = 0; u < 4; ++u) L[b * 4 + u] = __builtin_bit_cast(f32x4, __builtin_amdgcn_raw_buffer_load_b128(rx, vo + u * 64, so + b * 16 * KS * 4, 2));
                SCHED();
#pragma unroll
                for (int h = 0; h < 2; ++h)
#pragma unroll
                    for (int b = 0; b < 4; ++b) {
                        const f32x4 lo = L[b * 4 + 2 * h], hi = L[b * 4 + 2 * h + 1];
                        V[h * 4 + b] = (u32x4){pk(lo[0], lo[1]), pk(lo[2], lo[3]), pk(hi[0], hi[1]), pk(hi[2], hi[3])};
                    }
            } else {
                const __amdgpu_buffer_rsrc_t rw = mkrsrc((void*)a_w, KS * NN * 4);
                const unsigned vo = (fq * 4 * NN + fr * 4) * 4, so = (c * 64 * NN + g * 64) * 4;
#pragma unroll
                for (int u = 0; u < 4; ++u)
#pragma unroll
                    for (int j = 0; j < 4; ++j) L[u * 4 + j] = __builtin_bit_cast(f32x4, __builtin_amdgcn_raw_buffer_load_b128(rw, vo, so + (u * 16 + j) * NN * 4, 2));
                SCHED();
#pragma unroll
                for (int h = 0; h < 2; ++h)
#pragma unroll
                    for (int cc = 0; cc < 4; ++cc) {
                        const f32x4* l0 = &L[(2 * h) * 4]; const f32x4* l1 = &L[(2 * h + 1) * 4];
                        V[h * 4 + cc] = (u32x4){pk(l0[0][cc], l0[1][cc]), pk(l0[2][cc], l0[3][cc]), pk(l1[0][cc], l1[1][cc]), pk(l1[2][cc], l1[3][cc])};
                    }
            }
#pragma unroll
            for (int i = 0; i < 16; i += 5) asum += (__builtin_fabsf(L[i][0]) + __builtin_fabsf(L[i][1])) + (__builtin_fabsf(L[i][2]) + __builtin_fabsf(L[i][3]));
        }
        SCHED();
        call = rfl(callv) + 1u;
        raw = ws + WS_RAW + (call & 1u) * RAW_BYTES;
        if (have) {
            const __amdgpu_buffer_rsrc_t rexp = mkrsrc(raw, (unsigned)RAW_BYTES);
            const unsigned so = group_off(c, isw, g, rank);
#pragma unroll
            for (int v = 0; v < 8; ++v) __builtin_amdgcn_raw_buffer_store_b128(V[v], rexp, lane * 16 + v * 1024, so, 17);
            KEEP8(V);
        }
        if (q < PUSH_HEAD) {
            const unsigned vo = (unsigned)WS_RAW + (call & 1u) * (unsigned)RAW_BYTES + group_off(c, isw, g, rank) + lane * 16, vo2 = vo + 4096;
            unsigned long long pb[7];
#pragma unroll
            for (int i = 0; i < 7; ++i) {
                unsigned l = q + i; l = l % 7u; const unsigned p = (rank + 1u + l) & 7u;
                pb[i] = ((unsigned long long)(unsigned)__builtin_amdgcn_readlane(whi, p) << 32) | (unsigned)__builtin_amdgcn_readlane(wlo, p);
            }
            asm volatile("s_nop 4" : "+s"(pb[0]), "+s"(pb[1]), "+s"(pb[2]), "+s"(pb[3]), "+s"(pb[4]), "+s"(pb[5]), "+s"(pb[6]));
#pragma unroll
            for (int v = 0; v < 8; ++v)
#pragma unroll
                for (int i = 0; i < 7; ++i)
                    asm volatile("s_waitcnt vmcnt(%4)\n\tglobal_store_dwordx4 %0, %1, %2 offset:%3 sc0 sc1\n\ts_nop 1"
                                 :: "v"(v < 4 ? vo : vo2), "v"(V[v]), "s"(pb[i]), "i"((v & 3) * 1024), "i"(PUSH_WIN - 1) : "memory");
        }
        int si = __builtin_bit_cast(int, asum);
#define ROW_ADD(ctl) si = __builtin_bit_cast(int, __builtin_bit_cast(float, si) + __builtin_bit_cast(float, __builtin_amdgcn_update_dpp(0, si, ctl, 0xf, 0xf, true)))
        ROW_ADD(0x111); ROW_ADD(0x112); ROW_ADD(0x114); ROW_ADD(0x118);
        const float amean = __builtin_bit_cast(float, __builtin_amdgcn_readlane(si, 15)) * (1.0f / 256.f);
        if (have && q >= DEEP_FROM) {
            unsigned mx = 0;
#pragma unroll
            for (int v = 0; v < 8; ++v)
#pragma unroll
                for (int k = 0; k < 4; ++k) mx = pk_max16(mx, V[v][k] & 0x7fff7fffu);
            mx = (mx & 0xffffu) > (mx >> 16) ? mx & 0xffffu : mx >> 16;
#define MAX_DPP(ctl, rm) do { const unsigned y = __builtin_amdgcn_update_dpp(0u, mx, ctl, rm, 0xf, true); mx = y > mx ? y : mx; } while (0)
            MAX_DPP(0x111, 0xf); MAX_DPP(0x112, 0xf); MAX_DPP(0x114, 0xf); MAX_DPP(0x118, 0xf); MAX_DPP(0x142, 0xa); MAX_DPP(0x143, 0xc);
            const unsigned top0 = pk(amean * 5.7f, 0.f) & 0x7fffu, top1 = __builtin_amdgcn_readlane(mx, 63) + 1u;
            const unsigned base = rfl(((top0 > top1 ? top0 : top1) - 4096u) & 0x7fffu), base2 = base * 0x00010001u;
            LAS unsigned* const tl = (LAS unsigned*)(lds + 4096 + wid * 1024);
            tl[lane] = 0; tl[64 + lane] = 0;
            unsigned C[32];
            unsigned long long acc0 = 0, acc1 = 0;
            unsigned p0 = 0, p1 = 0;
#pragma unroll
            for (int i = 0; i < 32; ++i) {
                const unsigned t = pk_sub16(V[i >> 2][i & 3], base2), u = (t >> 10) & 0x001f001fu;
                C[i] = bfi(0x0c000c00u, pk_min16(u, 0x00030003u) << 10, t);
#pragma unroll
                for (int hh = 0; hh < 2; ++hh) {
                    const unsigned uu = hh ? u >> 16 : u & 0xffffu, r = 32u - uu;
                    const unsigned len = uu < 3 ? 0u : uu == 3 ? 3u : r + (r >= 3);
                    if (i < 16) { p0 += len; acc0 |= (unsigned long long)(len != 0) << ((p0 - 1) & 63); }
                    else { p1 += len; acc1 |= (unsigned long long)(len != 0) << ((p1 - 1) & 63); }
                }
            }
#pragma unroll
            for (int j = 0; j < 6; ++j) {
#pragma unroll
                for (int k = 0; k < 4; ++k) C[4 * j + k] = bfi(0x70007000u, C[26 + j] << (12 - 3 * k), C[4 * j + k]);
                C[24 + j / 3] = bfi(0x10001000u << (j % 3), C[26 + j] >> (3 - j % 3), C[24 + j / 3]);
            }
            const __amdgpu_buffer_rsrc_t rdp = mkrsrc(ws + WS_DEEP + (call & 1u) * PACK_BYTES, (unsigned)PACK_BYTES);
            u32x4 W[6];
#pragma unroll
            for (int v = 0; v < 6; ++v) { W[v] = (u32x4){C[4 * v], C[4 * v + 1], C[4 * v + 2], C[4 * v + 3]}; __builtin_amdgcn_raw_buffer_store_b128(W[v], rdp, lane * 16 + v * 1024, deep_off(q), 17); }
            KEEP4(W); KEEP1(W[4]); KEEP1(W[5]);
            typedef unsigned u32x2 __attribute__((ext_vector_type(2)));
            __builtin_amdgcn_raw_buffer_store_b64((u32x2){C[24], C[25]}, rdp, 6144 + lane * 8, deep_off(q), 17);
            const unsigned len = p0 + p1, end = wave_scan(len), total = __builtin_amdgcn_readlane(end, 63);
            const unsigned mode = __any(p0 > 64 || p1 > 64) || total > 444 * 8 ? 2u : total > 316 * 8 ? 1u : 0u;
            LDS_FENCE();
            if (mode < 2) {
                ((LAS unsigned char*)tl)[lane] = (unsigned char)len;
#pragma unroll
                for (int hh = 0; hh < 2; ++hh) {
                    const unsigned o = end - len + (hh ? p0 : 0u), sh = o & 31u;
                    const unsigned long long a = hh ? acc1 : acc0;
                    LAS unsigned* dp = tl + 17 + (o >> 5);
                    __hip_atomic_fetch_or(dp, (unsigned)a << sh, __ATOMIC_RELAXED, __HIP_MEMORY_SCOPE_WORKGROUP);
                    __hip_atomic_fetch_or(dp + 1, (unsigned)((a << sh) >> 32), __ATOMIC_RELAXED, __HIP_MEMORY_SCOPE_WORKGROUP);
                    __hip_atomic_fetch_or(dp + 2, (unsigned)((a >> 32) >> (32 - sh)), __ATOMIC_RELAXED, __HIP_MEMORY_SCOPE_WORKGROUP);
                }
            }
            if (lane == 0) tl[16] = base | (mode << 16);
            LDS_FENCE();
            __builtin_amdgcn_raw_buffer_store_b64((u32x2){tl[2 * lane], tl[2 * lane + 1]}, rdp, 7168 + lane * 8, deep_off(q), 17);
        }
        if (have && q >= RAW_HEAD && (q < DEEP_FROM || q >= 768)) {
            const float top = amean * 4.17f;
            int bb = (int)(pk(top, top) & 0x7fffu) - 8192;
            bb = bb < 0 ? 0 : bb;
            const unsigned b6 = rfl((unsigned)bb >> 6), b2 = b6 * 0x00400040u;
            LAS unsigned short* const rec16 = (LAS unsigned short*)(lds + wid * 512);
            ((LAS unsigned*)rec16)[lane] = 0;
            LDS_FENCE();
            unsigned E[5] = {0, 0, 0, 0, 0};
#pragma unroll
            for (int k = 0; k < 4; ++k) {
                const unsigned t7 = pk_sub16(V[7][k], b2), c7 = bfi(0x1fff1fffu, t7, t7 >> 2);
                E[4] = bfi(0x00030003u << (2 * k), t7 >> (13 - 2 * k), E[4]);
#pragma unroll
                for (int v = 0; v < 7; ++v) {
                    const unsigned t = pk_sub16(V[v][k], b2);
                    E[k] = bfi(0x00030003u << (2 * v), t >> (13 - 2 * v), E[k]);
                    V[v][k] = bfi(0x60006000u, c7 << (13 - 2 * v), t);
                }
            }
            const __amdgpu_buffer_rsrc_t rpak = mkrsrc(ws + WS_PACK + (call & 1u) * PACK_BYTES, (unsigned)PACK_BYTES);
#pragma unroll
            for (int v = 0; v < 7; ++v) __builtin_amdgcn_raw_buffer_store_b128(V[v], rpak, lane * 16 + v * 1024, pack_off(q), 17);
            KEEP8(V);
            unsigned cnt = 1;
#pragma unroll
            for (int a = 0; a < 5; ++a) {
                unsigned e = E[a];
                while (cnt <= 128) {
                    const unsigned long long m = __ballot(e != 0);
                    if (!m) break;
                    if (e != 0) {
                        const unsigned pos = __builtin_ctz(e) & 30u;
                        const unsigned slot = cnt + __builtin_amdgcn_mbcnt_hi((unsigned)(m >> 32), __builtin_amdgcn_mbcnt_lo((unsigned)m, 0));
                        if (slot < 128) rec16[slot] = (unsigned short)(((a * 64 + lane) << 6) | (pos << 1) | ((e >> pos) & 3u));
                        e &= ~(3u << pos);
                    }
                    cnt += __builtin_popcountll(m);
                }
            }
            if (lane == 0) rec16[0] = (unsigned short)(b6 | ((cnt <= 64 ? 0u : cnt <= 128 ? 1u : 2u) << 12));
            LDS_FENCE();
            __builtin_amdgcn_raw_buffer_store_b32(((LAS unsigned*)rec16)[lane], rpak, 7168 + lane * 4, pack_off(q), 17);
        }
        char* const pws = (char*)(((unsigned long long)(unsigned)whi << 32) | (unsigned)wlo);
        WAIT_V(0);
        const unsigned cofs = (unsigned)WS_CNT + (call & 1u) * 8192 + cnt_idx(c, isw, g, rank) * 4;
        if (wid == 0 && lane < NR && lane != rank) st_sys(pws + WS_SRCFLAG + rank * 2048 + gi * 4, call);
        if (have && lane == 0 && (isw || g < STRIP_G || g == STRIP_G + rank)) add_agent(ws + cofs, 1u);
        BAR();
        if (wid == 1) {
            if (lane < NR && lane != rank) st_sys(pws + WS_SRCFLAG + rank * 2048 + 1024 + gi * 4, call);
            if (lane == 0) st_agent(callp, call);
        }
    }
    STAMP_AT(1);

    const unsigned pm = (xcd & 1) * 6 + (loc >> 2), pn = (xcd >> 1) * 4 + (loc & 3);
    const bool light = pm == 11;
    const unsigned wr = wid >> 2, wc = wid & 3;
    unsigned voff0, voff1;
    {
        const unsigned ga = !light ? pm * 5 + wid : wid == 0 ? STRIP_G + rank : 55u;
        voff0 = (wid < 5 ? ga * GROUP_BYTES : W_OFF + (pn * 4 + wid - 5) * GROUP_BYTES) + lane * 16;
        voff1 = W_OFF + (pn * 4 + 3) * GROUP_BYTES + wid * 512 + (lane & 31) * 16;
    }
    const unsigned lds0 = (unsigned)(uintptr_t)lds;
    const unsigned a_rd = lane * 16 + wr * (light ? 4096 : 10240), b_rd = lane * 16 + 20480 + wc * 4096;
    const unsigned nth = (call + 1u) >> 1, cnt0 = (unsigned)WS_CNT + (call & 1u) * 8192;
    const unsigned needx = nth * (light ? 2u : 5u), needw = nth * 4u;
    const char* const cntx = ws + cnt0 + (pm * 64 + lane) * 4;
    const char* const cntw = ws + cnt0 + ((13 + pn) * 64 + lane) * 4;
    LAS unsigned long long* const ctrl = (LAS unsigned long long*)(lds + LDS_CTRL);

    unsigned long long avail = 0, taken = 0;

    auto ready = [&](unsigned cx, unsigned cwv) { return __builtin_amdgcn_ballot_w64(cx >= needx) & __builtin_amdgcn_ballot_w64(cwv >= needw) & ~taken; };
    auto poll = [&](unsigned cx, unsigned cwv) {
        unsigned long long m = ready(cx, cwv);
        if (!m) {
            const unsigned long long t0 = now();
            for (unsigned it = 1;; ++it) {
                cx = ld_agent(cntx); cwv = ld_agent(cntw);
                m = ready(cx, cwv);
                if (m) break;
                if (!(it & 15) && (rfl(ld_agent(ws + WS_TMO)) == call || now() - t0 > DEADLINE)) { if (lane == 0) st_agent(ws + WS_TMO, call); m = ~taken; break; }
                __builtin_amdgcn_s_sleep(2);
            }
        }
        if (lane == 0) *ctrl = m;
        WAIT_L(0);
    };
    const unsigned voffc = cnt0 + ((wid < 4 ? pm : 13 + pn) * 64 + (wid & 3) * 16 + (lane & 15)) * 4;
    auto snap = [&](unsigned which) {
        const bool on = ((avail & (avail - 1)) == 0) & (((~(taken | avail) >> ((wid & 3) * 16)) & 0xffffu) != 0);
        asm volatile("s_mov_b32 m0, %2\n\ts_mov_b64 exec, 0xffff\n\t"
                     "global_load_lds_dword %0, %1 sc1\n\t"
                     "s_mov_b64 exec, -1" :: "v"(voffc), "s"(ws + (on ? 0 : WS_ZERO)), "s"(lds0 + LDS_SNAP + which * 512 + wid * 64) : "memory");
    };
    auto fetch_mask = [&]() {
        const unsigned long long m = *(volatile LAS unsigned long long*)ctrl;
        avail = ((unsigned long long)rfl((unsigned)(m >> 32)) << 32) | rfl((unsigned)m);
    };
    const unsigned m0w = lds0 + wid * 4096, m0h = lds0 + 32768 + wid * 512;
    unsigned long long tbase = 0;
    auto stage = [&](unsigned slot) {
        asm volatile("s_mov_b32 m0, %3\n\ts_nop 0\n\t"
                     "global_load_lds_dwordx4 %0, %2 sc0 sc1\n\t"
                     "global_load_lds_dwordx4 %0, %2 offset:1024 sc0 sc1\n\t"
                     "global_load_lds_dwordx4 %0, %2 offset:2048 sc0 sc1\n\t"
                     "global_load_lds_dwordx4 %0, %2 offset:3072 sc0 sc1\n\t"
                     "s_mov_b32 m0, %4\n\ts_mov_b32 exec_hi, 0\n\t"
                     "global_load_lds_dwordx4 %1, %2 sc0 sc1\n\t"
                     "s_mov_b32 exec_hi, -1" :: "v"(voff0), "v"(voff1), "s"(tbase), "s"(m0w + slot), "s"(m0h + slot) : "memory");
    };
    auto pick = [&]() {
        const unsigned t = __builtin_ctzll(avail);
        avail &= avail - 1; taken |= 1ull << t;
        tbase = (unsigned long long)raw + t * TILE_BYTES;
    };
    constexpr unsigned NSTEP = 128;
#define STEP_DIAG
    auto k_loop = [&](auto& acc, auto nblk) {
        constexpr int NB = nblk.value;
        ((LAS unsigned*)(lds + LDS_SNAP))[tid] = 0;
        if (tid < 128) ((LAS unsigned*)(lds + LDS_SNAP + 2048))[tid] = 0;
        WAIT_V(0); WAIT_L(0);
        BAR();
#define READ_FRAGS(off) \
            _Pragma("unroll") for (int n = 0; n < 4; ++n) Bt[n] = *(const LAS f16x8*)(pb + (off) + n * 1024); \
            _Pragma("unroll") for (int m = 0; m < NB; ++m) At[m] = *(const LAS f16x8*)(pa + (off) + m * 1024)
#define READ_SOME(off, k0, k1) do { _Pragma("unroll") for (int k = (k0); k < (k1) && k < NB + 4; ++k) { \
                if (k < 4) Bt[k] = *(const LAS f16x8*)(pb + (off) + k * 1024); else At[k - 4] = *(const LAS f16x8*)(pa + (off) + (k - 4) * 1024); } SCHED(); } while (0)
#define READ_AND_STAGE(off, slot) do { constexpr int RS = (NB + 4 + 5) / 6; \
            READ_SOME(off, 0, RS); \
            asm volatile("s_mov_b32 m0, %2\n\ts_nop 0\n\tglobal_load_lds_dwordx4 %0, %1 sc0 sc1" :: "v"(voff0), "s"(tbase), "s"(m0w + (slot)) : "memory"); \
            READ_SOME(off, RS, 2 * RS); \
            asm volatile("global_load_lds_dwordx4 %0, %1 offset:1024 sc0 sc1" :: "v"(voff0), "s"(tbase) : "memory"); \
            READ_SOME(off, 2 * RS, 3 * RS); \
            asm volatile("global_load_lds_dwordx4 %0, %1 offset:2048 sc0 sc1" :: "v"(voff0), "s"(tbase) : "memory"); \
            READ_SOME(off, 3 * RS, 4 * RS); \
            asm volatile("global_load_lds_dwordx4 %0, %1 offset:3072 sc0 sc1" :: "v"(voff0), "s"(tbase) : "memory"); \
            READ_SOME(off, 4 * RS, 6 * RS); \
            asm volatile("s_mov_b32 m0, %2\n\ts_mov_b32 exec_hi, 0\n\tglobal_load_lds_dwordx4 %0, %1 sc0 sc1\n\ts_mov_b32 exec_hi, -1" :: "v"(voff1), "s"(tbase), "s"(m0h + (slot)) : "memory"); \
        } while (0)
#define MFMAS \
            _Pragma("unroll") for (int m = 0; m < NB; ++m) _Pragma("unroll") for (int n = 0; n < 4; ++n) \
                acc[m][n] = __builtin_amdgcn_mfma_f32_16x16x32_f16(At[m], Bt[n], acc[m][n], 0, 0, 0)
#define COMPUTE do { WAIT_L(0); SCHED(); BAR(); __builtin_amdgcn_s_setprio(1); MFMAS; __builtin_amdgcn_s_setprio(0); SCHED(); BAR(); SCHED(); } while (0)
#define TILE_HEAD \
            STEP_DIAG \
            const unsigned cb = (st & 2) * SLOT_BYTES, ob = 2 * SLOT_BYTES - cb; \
            const LAS char* pa = lds + cb + a_rd; const LAS char* pb = lds + cb + b_rd; \
            const LAS unsigned* sp = (const LAS unsigned*)(lds + LDS_SNAP + (st & 2) * 512) + lane;       \
            f16x8 At[NB], Bt[4];
        unsigned st = 0;
        unsigned ahead = 0;
        for (;;) {
            if (ahead == 0) {
                if (wr == 0) {
                    if (wid == 0) { WAIT_V(0); const LAS unsigned* fp = (const LAS unsigned*)(lds + LDS_SNAP + 2048) + lane; poll(fp[0], fp[64]); }
                    BAR();
                } else if (st == 0) BAR();
                fetch_mask();
                const unsigned cb = (st & 2) * SLOT_BYTES;
                pick(); stage(cb); tbase += 4096; stage(cb + SLOT_BYTES);
                if (avail) { pick(); stage(2 * SLOT_BYTES - cb); ahead = 3; WAIT_V(10); } else { ahead = 2; WAIT_V(5); }
                BAR();
                if (wr == 1) BAR();
            }
            while (ahead == 3) {
                TILE_HEAD
                snap((st + 2) & 3);
                tbase += 4096;
                READ_AND_STAGE(0, ob + SLOT_BYTES);
                const unsigned sx = sp[0], sw = sp[64];
                WAIT_V(11); WAIT_L(0); SCHED();
                BAR();
                __builtin_amdgcn_s_setprio(1);
                const unsigned long long m2 = avail | ready(sx, sw), low = m2 & (0ull - m2);
                tbase = (unsigned long long)raw + (unsigned)__builtin_ctzll(m2 | (1ull << 63)) * TILE_BYTES;
                avail = m2 ^ low; taken |= low;
                ahead = m2 ? 3 : 2;
                MFMAS;
#pragma unroll
                for (int k = 0; k < 2; ++k) { __builtin_amdgcn_sched_group_barrier(0x8, 1, 0); __builtin_amdgcn_sched_group_barrier(0x2, 1, 0); }
#pragma unroll
                for (int k = 0; k < 12; ++k) { __builtin_amdgcn_sched_group_barrier(0x8, 1, 0); __builtin_amdgcn_sched_group_barrier(0x4, 2, 0); }
                __builtin_amdgcn_sched_group_barrier(0x8, 4 * NB - 14, 0);
                asm volatile("" : "+s"(tbase), "+s"(avail), "+s"(taken), "+s"(ahead));
                __builtin_amdgcn_s_setprio(0); SCHED();
                BAR(); SCHED();
                snap((st + 3) & 3);
                if (ahead == 3) { READ_AND_STAGE(SLOT_BYTES, cb); WAIT_V(11); } else { READ_FRAGS(SLOT_BYTES); WAIT_V(6); }
                COMPUTE;
                st += 2;
            }
            {
                TILE_HEAD
                const unsigned sx = sp[0], sw = sp[64];
                READ_FRAGS(0);
                snap((st + 2) & 3);
                avail = ready(sx, sw);
                const bool late = avail != 0;
                if (late) { pick(); stage(ob); tbase += 4096; stage(ob + SLOT_BYTES); WAIT_V(11); } else WAIT_V(1);
                COMPUTE;
                const unsigned sx1 = sp[128], sw1 = sp[192];
                READ_FRAGS(SLOT_BYTES);
                snap((st + 3) & 3);
                avail |= ready(sx1, sw1);
                if (late) {
                    if (avail) { pick(); stage(cb); ahead = 3; WAIT_V(11); } else { ahead = 2; WAIT_V(6); }
                } else if (avail) {
                    pick(); stage(ob); tbase += 4096; stage(ob + SLOT_BYTES);
                    if (avail) { pick(); stage(cb); ahead = 3; WAIT_V(10); } else { ahead = 2; WAIT_V(5); }
                } else {
                    ahead = 0;
                    WAIT_V(1);
                    if (st + 2 == NSTEP) {
                        asm volatile("s_mov_b32 m0, %2\n\ts_mov_b64 exec, 1\n\tglobal_load_lds_dword %0, %1 sc1\n\ts_mov_b64 exec, -1"
                                     :: "v"((unsigned)WS_TMO), "s"(ws), "s"(lds0 + LDS_CTRL + 16 + wid * 4) : "memory");
                    } else if (wid == 0) {
                        asm volatile("s_mov_b32 m0, %2\n\ts_nop 0\n\tglobal_load_lds_dword %0, off sc1\n\t"
                                     "s_mov_b32 m0, %3\n\ts_nop 0\n\tglobal_load_lds_dword %1, off sc1"
                                     :: "v"(cntx), "v"(cntw), "s"(lds0 + LDS_SNAP + 2048), "s"(lds0 + LDS_SNAP + 2304) : "memory");
                    }
                }
                COMPUTE;
                st += 2;
            }
            if (st == NSTEP) break;
        }
        if (wr == 0) BAR();
        STAMP_AT(3);
        WAIT_V(0);
    };

    const __amdgpu_buffer_rsrc_t rout = mkrsrc((void*)a_out, (unsigned)MM * NN * 4);
    const unsigned row0 = !light ? pm * 320 + wr * 160 : wr ? 55 * 64 : (STRIP_G + rank) * 64;
    const unsigned ovo = (fq * 4 * NN + fr * 4) * 4, oso = (row0 * NN + pn * 256 + wc * 64) * 4;
    float one = 1.0f;
    auto check_tmo = [&]() { one = rfl(*(const volatile LAS unsigned*)(lds + LDS_CTRL + 16 + wid * 4)) == call ? __builtin_nanf("") : 1.0f; };
    auto gelu_block = [&](const f32x4 (&a)[4], f32x4 (&o)[4]) {
        const float k1 = -2.0f * 0.7978845608028654f * 1.4426950408889634f, k2 = k1 * 0.044715f;
        f32x2 y[8], z[8];
#pragma unroll
        for (int i = 0; i < 8; ++i) { y[i] = (f32x2){a[i & 3][(i >> 2) * 2], a[i & 3][(i >> 2) * 2 + 1]}; z[i] = y[i] * y[i]; }
        SCHED();
#pragma unroll
        for (int i = 0; i < 8; ++i) z[i] = __builtin_elementwise_fma(z[i], (f32x2){k2, k2}, (f32x2){k1, k1});
        SCHED();
#pragma unroll
        for (int i = 0; i < 8; ++i) z[i] = z[i] * y[i];
        SCHED();
#pragma unroll
        for (int i = 0; i < 8; ++i) z[i] = (f32x2){__builtin_amdgcn_exp2f(z[i].x), __builtin_amdgcn_exp2f(z[i].y)};
        SCHED();
#pragma unroll
        for (int i = 0; i < 8; ++i) z[i] = z[i] + one;
        SCHED();
#pragma unroll
        for (int i = 0; i < 8; ++i) z[i] = (f32x2){__builtin_amdgcn_rcpf(z[i].x), __builtin_amdgcn_rcpf(z[i].y)};
        SCHED();
#pragma unroll
        for (int i = 0; i < 8; ++i) {
            float a0 = y[i].x * z[i].x, a1 = y[i].y * z[i].y;
            asm("" : "+v"(a0)); asm("" : "+v"(a1));
            o[(i >> 2) * 2][i & 3] = a0; o[(i >> 2) * 2 + 1][i & 3] = a1;
        }
    };
    if (light) {
        f32x4 acc[4][4];
#pragma unroll
        for (int m = 0; m < 4; ++m)
#pragma unroll
            for (int n = 0; n < 4; ++n) acc[m][n] = (f32x4){0.f, 0.f, 0.f, 0.f};
        k_loop(acc, std::integral_constant<int, 4>{});
        check_tmo();
        const __amdgpu_buffer_rsrc_t rxo = mkrsrc(ws + WS_XOUT + (call & 1u) * (1u << 20) + pn * 65536, 65536);
        unsigned oseq = 0;
        if (tid == 0) { unsigned z = 0; asm("" : "+v"(z)); oseq = __hip_atomic_fetch_add((GAS unsigned*)(ws + WS_OSEQ) + z, 1u, __ATOMIC_RELAXED, __HIP_MEMORY_SCOPE_AGENT); }
        if (wr == 1) BAR();
        f32x4 o[4][4];
#pragma unroll
        for (int m = 0; m < 4; ++m) {
            gelu_block(acc[m], o[m]);
            if (wr == 0) {
#pragma unroll
                for (int j = 0; j < 4; ++j) __builtin_amdgcn_raw_buffer_store_b128(__builtin_bit_cast(u32x4, o[m][j]), rxo, fq * 4096 + fr * 16, (m * 16 + j) * 1024 + wc * 256, 17);
            }
            KEEP4(o[m]);
            SCHED();
        }
        if (wr == 0) {
            WAIT_V0();
            BAR();
            if (wid == 0) {
                const unsigned k = rfl(oseq) & 15u;
                char* const pws = (char*)(((unsigned long long)(unsigned)whi << 32) | (unsigned)wlo);
                if (lane < NR && lane != rank) st_sys(pws + WS_OUTFLAG + rank * 128 + k * 4, call * 16 + pn);
            }
            STAMP_AT(9);
        }
#pragma unroll
        for (int m = 0; m < 4; ++m)
#pragma unroll
            for (int j = 0; j < 4; ++j) __builtin_amdgcn_raw_buffer_store_b128(__builtin_bit_cast(u32x4, o[m][j]), rout, ovo, oso + (m * 16 + j) * NN * 4, OUT_AUX);
        KEEP4(o[0]); KEEP4(o[1]); KEEP4(o[2]); KEEP4(o[3]);
    } else {
        f32x4 acc[10][4];
#pragma unroll
        for (int m = 0; m < 10; ++m)
#pragma unroll
            for (int n = 0; n < 4; ++n) acc[m][n] = (f32x4){0.f, 0.f, 0.f, 0.f};
        k_loop(acc, std::integral_constant<int, 10>{});
        check_tmo();
#pragma unroll
        for (int m = 0; m < 10; ++m) {
            f32x4 o[4];
            gelu_block(acc[m], o);
#pragma unroll
            for (int j = 0; j < 4; ++j) __builtin_amdgcn_raw_buffer_store_b128(__builtin_bit_cast(u32x4, o[j]), rout, ovo, oso + (m * 16 + j) * NN * 4, OUT_AUX);
            KEEP4(o);
            SCHED();
        }
    }
}

}

extern "C" void kernel_launch(int rank, int nranks, ncclComm_t comm,
                              hipStream_t stream,
                              void* const* d_in, const int* in_sizes, int n_in,
                              void* d_out, int out_size,
                              const int* in_dtypes, int out_dtype,
                              void* const* d_out_all,
                              void* const* d_ws_all, size_t ws_bytes) {
    (void)comm; (void)in_dtypes; (void)out_dtype; (void)d_out_all;
    if (nranks != NR || n_in != 2 || (unsigned)rank >= (unsigned)NR || ws_bytes < WS_END || out_size != MM * NN ||
        in_sizes[0] != MM * KS || in_sizes[1] != KS * NN) {
        fprintf(stderr, "kernel_launch: unexpected problem shape\n");
        return;
    }
    HIP_CHECK(hipFuncSetAttribute((const void*)fused, hipFuncAttributeMaxDynamicSharedMemorySize, LDS_BYTES));
    Params P;
    P.x = (const float*)d_in[0]; P.w = (const float*)d_in[1]; P.out = (float*)d_out;
    for (int i = 0; i < NR; ++i) P.ws[i] = (char*)d_ws_all[i];
    P.self = (char*)d_ws_all[rank]; P.rank = rank; P.pad = 0;
    hipLaunchKernelGGL(fused, dim3(NWG), dim3(NTHREADS), LDS_BYTES, stream, P);
    HIP_CHECK(hipGetLastError());
}
```
